```python
import numpy as np

kernel_name = "gpu_a2a_gemm_m4096_k4096_n16384_bf16_silu"


ABSMAX_RTOL = 0.008

COMPUTE_DTYPE = "bfloat16"
EPILOGUE = "silu"
IN_QUANT = None
OUT_QUANT = None
IN_DTYPES = {"x": "bfloat16", "w_mat": "bfloat16"}
WORLD_SIZE = 8

def _to_bf16(x):
    u = np.ascontiguousarray(x, dtype=np.float32).view(np.uint32)
    return ((u + 0x7FFF + ((u >> 16) & 1)) & 0xFFFF0000).view(np.float32)

def setup_inputs():

    x = _to_bf16(np.random.randn(4096, 4096).astype(np.float32))
    w_mat = _to_bf16((np.random.randn(4096, 16384) / np.sqrt(4096)).astype(np.float32))
    return {"x": x, "w_mat": w_mat}

def _epilogue(y):
    return (y / (1.0 + np.exp(-np.clip(y, -60.0, 60.0)))).astype(np.float32)

def reference(x, w_mat):
    y = (x @ w_mat).astype(np.float32)
    return _epilogue(y)
```

```cpp
#include <hip/hip_runtime.h>
#include <rccl/rccl.h>
#include <cstdio>

#define HIP_CHECK(expr) do { hipError_t _e = (expr); if (_e != hipSuccess) { \
    fprintf(stderr, "HIP error %s at %s:%d\n", hipGetErrorString(_e), __FILE__, __LINE__); \
    return; } } while (0)

typedef short  s16x8 __attribute__((ext_vector_type(8)));
typedef short  s16x4 __attribute__((ext_vector_type(4)));
typedef float  f32x4 __attribute__((ext_vector_type(4)));
typedef float  f32x2 __attribute__((ext_vector_type(2)));
typedef __bf16 bf16x2 __attribute__((ext_vector_type(2)));
typedef unsigned u32x4 __attribute__((ext_vector_type(4)));
typedef unsigned u32x2 __attribute__((ext_vector_type(2)));
typedef __attribute__((address_space(3))) char lds_char;

constexpr int NR = 8;
constexpr int MS = 512, KS = 512, KT = 4096, NT = 16384;
constexpr int STAGE_B = 49152, A_B = 32768;
constexpr int LDS_B = 3 * STAGE_B;
constexpr size_t WS_CNT = 0;
constexpr size_t WS_TMO = 4096;
constexpr size_t WS_FLG = 8192;
constexpr size_t WS_STG = 1u << 20;
constexpr size_t SLOT_B = (size_t)NR * MS * KS * 2;
constexpr unsigned NSLOT = 8;
constexpr unsigned long long DEADLINE_TICKS = 40000000ull;

#define WAITV(n) asm volatile("s_waitcnt vmcnt(" #n ")" ::: "memory")
#define WAITL0() asm volatile("s_waitcnt lgkmcnt(0)" ::: "memory")
#define BAR() asm volatile("s_barrier" ::: "memory")
#define SCHED() __builtin_amdgcn_sched_barrier(0)

__device__ __forceinline__ void glds(unsigned voff, const char* sbase, unsigned ldsaddr) {
  asm volatile("s_mov_b32 m0, %2\n\ts_nop 0\n\tglobal_load_lds_dwordx4 %0, %1" :: "v"(voff), "s"(sbase), "s"(ldsaddr) : "memory");
}
__device__ __forceinline__ void glds128(unsigned voff, const char* sbase, unsigned ldsaddr) {
  asm volatile("s_mov_b32 m0, %2\n\ts_nop 0\n\tglobal_load_lds_dwordx4 %0, %1 offset:128" :: "v"(voff), "s"(sbase), "s"(ldsaddr - 128u) : "memory");
}

__device__ __forceinline__ float silu_f(float y) {
  float c = __builtin_amdgcn_fmed3f(y, -60.f, 60.f);
  float e = __builtin_amdgcn_exp2f(c * -1.4426950408889634f);
  return y * __builtin_amdgcn_rcpf(1.f + e);
}

__global__ __launch_bounds__(512, 2)
void a2a_gemm_silu(const char* __restrict__ x, const char* __restrict__ w, char* __restrict__ out,
                   char* const* __restrict__ ws_all, int rank, int mode) {
  extern __shared__ __attribute__((aligned(1024))) char smem[];
  lds_char* lds = (lds_char*)smem;
  const unsigned lds0 = (unsigned)(size_t)lds;
  const int tid = threadIdx.x, lane = tid & 63;
  const int wv = __builtin_amdgcn_readfirstlane(tid >> 6);
  const int g = wv >> 2, wg = wv & 3, wm = wg >> 1, wn = wg & 1;
  const int bid = blockIdx.x;
  const int xcd = bid & 7, jj = bid >> 3, mh = jj & 1, ntile = xcd * 16 + (jj >> 1);
  const bool comm = (mode & 1) != 0;
  char* const ws = ws_all[rank];

  const int myround = (comm && bid < 224) ? bid / 28 : -100;
  const int prem = bid % 28, pdest = (rank + 8 - (prem / 4 + 1)) & 7, pq = prem & 3;

  unsigned cntv; u32x4 px0, px1;
  const unsigned voffP = (unsigned)((tid >> 3) * 1024 + (tid & 7) * 16);
  {
    const char* cp = ws + WS_CNT + bid * 4;
    asm volatile("global_load_dword %0, %1, off sc0 sc1" : "=v"(cntv) : "v"(cp) : "memory");
    const int rr = (myround >= 0) ? myround : 0;
    const char* xp = x + (size_t)(pdest * MS + pq * 128) * 1024 + rr * 128;
    asm volatile("global_load_dwordx4 %0, %1, %2" : "=v"(px0) : "v"(voffP), "s"(xp) : "memory");
    asm volatile("global_load_dwordx4 %0, %1, %2" : "=v"(px1) : "v"(voffP), "s"(xp + 65536) : "memory");
  }

  const int r8 = lane >> 3;
  const unsigned voffA0 = (unsigned)((64 * wg + r8) * 1024 + (((lane & 7) ^ (r8 >> 1)) * 16));
  const unsigned voffA1 = (voffA0 ^ 64u) + 8192u;
  const int rb = (lane >> 2) & 7, sub = lane >> 5, slot4 = lane & 3;
  const unsigned voffB0 = (unsigned)((16 * wg + rb) * 32768 + sub * 64 + ((slot4 ^ (rb >> 2)) * 16));
  const unsigned voffB1 = (unsigned)((16 * wg + 8 + rb) * 32768 + sub * 64 + ((slot4 ^ (2 | (rb >> 2))) * 16));
  const char* const xloc = x + (size_t)(rank * MS + mh * 256) * 1024;
  const char* const wcol = w + (size_t)ntile * 256;

  auto stage_tile = [&](const char* abase, const char* bbase, unsigned sbyte) {
    const unsigned la = lds0 + sbyte + wg * 8192;
#pragma unroll
    for (int j = 0; j < 4; ++j) {
      glds(voffA0, abase + j * 16384, la + j * 2048);
      glds(voffA1, abase + j * 16384, la + j * 2048 + 1024);
    }
    const unsigned lb = lds0 + sbyte + A_B + wg * 4096;
    glds(voffB0, bbase, lb);
    glds128(voffB0, bbase, lb + 1024);
    glds(voffB1, bbase, lb + 2048);
    glds128(voffB1, bbase, lb + 3072);
  };

  stage_tile(xloc + g * 128, wcol + (size_t)(rank * KS + g * 64) * 32768, g * STAGE_B);
  asm volatile("s_waitcnt vmcnt(12)" : "+v"(cntv), "+v"(px0), "+v"(px1) :: "memory");
  const unsigned call = __builtin_amdgcn_readfirstlane(cntv) + 1u;
  const size_t slot_off = WS_STG + (size_t)(call % NSLOT) * SLOT_B;
  const char* const stg = ws + slot_off + (size_t)mh * 256 * 1024;
  const char* const flg = ws + WS_FLG + mh * 128;
  unsigned* const tmo = (unsigned*)(ws + WS_TMO);

  char* pdst = nullptr; char* pflag = nullptr;
  if (myround >= 0) {
    char* wsd = ws_all[(unsigned)pdest < (unsigned)NR ? pdest : rank];
    pdst = wsd + slot_off + (size_t)rank * (MS * KS * 2) + (size_t)(pq * 128) * 1024 + myround * 128;
    pflag = wsd + WS_FLG + ((rank * 8 + myround) * 2 + (pq >> 1)) * 128 + ((pq & 1) * 2 + g) * 4;
  }
  auto push = [&]() {
    asm volatile("global_store_dwordx4 %0, %1, %2 sc0 sc1\n\ts_nop 1" :: "v"(voffP), "v"(px0), "s"(pdst) : "memory");
    asm volatile("global_store_dwordx4 %0, %1, %2 sc0 sc1\n\ts_nop 1" :: "v"(voffP), "v"(px1), "s"(pdst + 65536) : "memory");
  };
  if (myround == 0) { push(); WAITV(2); } else { WAITV(0); }
  BAR();
  if (g == 1) BAR();

  const int fr = lane & 15, fq = lane >> 4;
  unsigned rdA = lds0 + g * STAGE_B + (unsigned)((128 * wm + fr) * 128 + ((fq ^ (fr >> 1)) * 16));
  unsigned rdB = lds0 + g * STAGE_B + (unsigned)(A_B + 2048 * fq + 1024 * wn + 64 * ((lane & 15) >> 2) + 8 * (lane & 1)
                                                 + 16 * (((lane >> 1) & 1) ^ ((fq & 1) << 1)));

  f32x4 acc[8][4];
#pragma unroll
  for (int m = 0; m < 8; ++m)
#pragma unroll
    for (int n = 0; n < 4; ++n) acc[m][n] = f32x4{0.f, 0.f, 0.f, 0.f};

  s16x8 fa[8][2], fb[4][2];
  unsigned pollv = 0;
  unsigned sread = g * STAGE_B;
  unsigned swrite = (g == 0) ? 2 * STAGE_B : 0;
  const bool leader = (wg == 0);

  for (int i = 0; i < 32; ++i) {
    const int T = 2 * i + g;
    {
      asm volatile("" : "+v"(rdA), "+v"(rdB));
      lds_char* pa0 = (lds_char*)(size_t)rdA;
      lds_char* pa1 = (lds_char*)(size_t)(rdA ^ 64u);
      lds_char* pbb[2][2] = {{(lds_char*)(size_t)rdB, (lds_char*)(size_t)(rdB ^ 16u)},
                             {(lds_char*)(size_t)(rdB ^ 32u), (lds_char*)(size_t)(rdB ^ 48u)}};
#pragma unroll
      for (int n = 0; n < 4; ++n)
#pragma unroll
        for (int ks = 0; ks < 2; ++ks) {
          lds_char* pb0 = pbb[n & 1][0] + (8192 * ks + 512 * (n >> 1));
          lds_char* pb1 = pbb[n & 1][1] + (8192 * ks + 512 * (n >> 1) + 256);
          s16x4 lo = __builtin_amdgcn_ds_read_tr16_b64_v4i16((__attribute__((address_space(3))) s16x4*)pb0);
          s16x4 hi = __builtin_amdgcn_ds_read_tr16_b64_v4i16((__attribute__((address_space(3))) s16x4*)pb1);
          fb[n][ks] = __builtin_shufflevector(lo, hi, 0, 1, 2, 3, 4, 5, 6, 7);
        }
#pragma unroll
      for (int m = 0; m < 8; ++m) {
        fa[m][0] = *(__attribute__((address_space(3))) s16x8*)(pa0 + 2048 * m);
        fa[m][1] = *(__attribute__((address_space(3))) s16x8*)(pa1 + 2048 * m);
      }
      SCHED();
      const int Tn = T + 2;
      if (Tn < 64) {
        int src, rd;
        if (Tn < 8) { src = rank; rd = Tn; }
        else { const int u = Tn - 8; rd = u / 7; src = (rank + (u - rd * 7) + 1) & 7; }
        const char* abase = (Tn < 8) ? xloc + rd * 128 : stg + (size_t)src * (MS * KS * 2) + rd * 128;
        stage_tile(abase, wcol + (size_t)(src * KS + rd * 64) * 32768, swrite);
      }
    }
    const int Tp = T + 4;
    const bool dopoll = comm && leader && Tp >= 8 && Tp < 64;
    const char* pollp = flg;
    if (dopoll) {
      const int u = Tp - 8, rd = u / 7, src = (rank + (u - rd * 7) + 1) & 7;
      pollp = flg + (src * 8 + rd) * 256;
      asm volatile("v_and_b32 %0, 48, %1\n\tv_lshrrev_b32 %0, 2, %0\n\tglobal_load_dword %0, %0, %2 sc0 sc1" : "=&v"(pollv) : "v"(voffP), "s"(pollp) : "memory");
    }
    int pend = 0;
    if (i == myround - 1) { push(); pend = 2; }
    if (i == myround + 1 && leader) {
      unsigned t0v, t1v;
      asm volatile("v_mov_b32 %0, 0\n\tv_mov_b32 %1, %3\n\tglobal_store_dword %0, %1, %2 sc0 sc1\n\ts_nop 1" : "=&v"(t0v), "=&v"(t1v) : "s"(pflag), "s"(call) : "memory");
      pend = 1;
    }
    WAITL0();
    BAR();
    SCHED();
    __builtin_amdgcn_s_setprio(1);
#pragma unroll
    for (int ks = 0; ks < 2; ++ks)
#pragma unroll
      for (int m = 0; m < 8; ++m) {
        if (ks == 1 && m == 6) {
          SCHED();
          if (pend == 0) WAITV(0); else if (pend == 1) WAITV(1); else WAITV(2);
          if (dopoll) {
            asm volatile("" : "+v"(pollv));
            if (__builtin_amdgcn_ballot_w64(pollv < call) != 0ull) {
              unsigned long long t0 = __builtin_amdgcn_s_memrealtime();
              for (;;) {
                unsigned a = __hip_atomic_load((const unsigned*)pollp + (lane & 3), __ATOMIC_RELAXED, __HIP_MEMORY_SCOPE_SYSTEM);
                if (__builtin_amdgcn_ballot_w64(a < call) == 0ull) break;
                if (__hip_atomic_load(tmo, __ATOMIC_RELAXED, __HIP_MEMORY_SCOPE_AGENT) == call ||
                    __builtin_amdgcn_s_memrealtime() - t0 > DEADLINE_TICKS) {
                  __hip_atomic_store(tmo, call, __ATOMIC_RELAXED, __HIP_MEMORY_SCOPE_AGENT);
                  break;
                }
                __builtin_amdgcn_s_sleep(1);
              }
              WAITV(0);
            }
          }
          BAR();
          SCHED();
        }
#pragma unroll
        for (int n = 0; n < 4; ++n)
          acc[m][n] = __builtin_amdgcn_mfma_f32_16x16x32_bf16(fb[n][ks], fa[m][ks], acc[m][n], 0, 0, 0);
      }
    __builtin_amdgcn_s_setprio(0);
    SCHED();
    const unsigned nx = (swrite >= STAGE_B) ? swrite - STAGE_B : swrite + 2 * STAGE_B;
    rdA += swrite - sread; rdB += swrite - sread;
    sread = swrite; swrite = nx;
  }
  if (g == 0) BAR();

  WAITV(0);
  const bool bad = comm && (__hip_atomic_load(tmo, __ATOMIC_RELAXED, __HIP_MEMORY_SCOPE_AGENT) == call);
  {
    lds_char* mine = lds + wv * 16384 + lane * 16;
#pragma unroll
    for (int m = 0; m < 4; ++m)
#pragma unroll
      for (int n = 0; n < 4; ++n) {
        f32x4 v = (g == 0) ? acc[m + 4][n] : acc[m][n];
        *(__attribute__((address_space(3))) f32x4*)(mine + (m * 4 + n) * 1024) = v;
      }
  }
  WAITL0();
  BAR();
  lds_char* theirs = lds + (wv ^ 4) * 16384;
  f32x4 fin[4][4];
#pragma unroll
  for (int m = 0; m < 4; ++m)
#pragma unroll
    for (int n = 0; n < 4; ++n) {
      f32x4 o = *(__attribute__((address_space(3))) f32x4*)(theirs + lane * 16 + (m * 4 + n) * 1024);
      f32x4 v = (g == 0) ? acc[m][n] : acc[m + 4][n];
      fin[m][n] = v + o;
    }
  WAITL0();
  SCHED();
#pragma unroll
  for (int m = 0; m < 4; ++m)
#pragma unroll
    for (int n = 0; n < 4; ++n) {
      f32x4 v = fin[m][n];
      bf16x2 lo = __builtin_convertvector(f32x2{silu_f(v[0]), silu_f(v[1])}, bf16x2);
      bf16x2 hi = __builtin_convertvector(f32x2{silu_f(v[2]), silu_f(v[3])}, bf16x2);
      u32x2 pk = u32x2{__builtin_bit_cast(unsigned, lo), __builtin_bit_cast(unsigned, hi)};
      if (bad) pk = u32x2{0x7fc07fc0u, 0x7fc07fc0u};
      const int row = 16 * m + fr, c = 2 * n + (fq >> 1);
      *(__attribute__((address_space(3))) u32x2*)(theirs + row * 128 + ((c ^ (row & 7)) * 16) + (fq & 1) * 8) = pk;
    }
  WAITL0();
  SCHED();
  {
    char* op = out + (size_t)(mh * 256 + wm * 128 + g * 64 + r8) * 32768 + (size_t)ntile * 256 + wn * 128 + (lane & 7) * 16;
#pragma unroll
    for (int i = 0; i < 8; ++i) {
      const int row = i * 8 + r8;
      u32x4 v = *(__attribute__((address_space(3))) u32x4*)(theirs + row * 128 + (((lane & 7) ^ (row & 7)) * 16));
      *(u32x4*)(op + (size_t)i * 8 * 32768) = v;
    }
  }
  if (tid == 0) __hip_atomic_store((unsigned*)(ws + WS_CNT + bid * 4), call, __ATOMIC_RELAXED, __HIP_MEMORY_SCOPE_AGENT);
}

extern "C" void kernel_launch(int rank, int nranks, ncclComm_t comm,
                              hipStream_t stream,
                              void* const* d_in, const int* in_sizes, int n_in,
                              void* d_out, int out_size,
                              const int* in_dtypes, int out_dtype,
                              void* const* d_out_all,
                              void* const* d_ws_all, size_t ws_bytes) {
  if (nranks != NR || ws_bytes < WS_STG + NSLOT * SLOT_B) { fprintf(stderr, "unexpected configuration\n"); return; }
  static bool init = false;
  if (!init) {
    HIP_CHECK(hipFuncSetAttribute((const void*)a2a_gemm_silu, hipFuncAttributeMaxDynamicSharedMemorySize, LDS_B));
    init = true;
  }
  hipLaunchKernelGGL(a2a_gemm_silu, dim3(256), dim3(512), LDS_B, stream,
                     (const char*)d_in[0], (const char*)d_in[1], (char*)d_out,
                     (char* const*)d_ws_all, rank, 1);
  HIP_CHECK(hipGetLastError());
}
```

```cpp
#include <hip/hip_runtime.h>
#include <rccl/rccl.h>
#include <cstdio>

#define HIP_CHECK(expr) do { hipError_t _e = (expr); if (_e != hipSuccess) { \
    fprintf(stderr, "HIP error %s at %s:%d\n", hipGetErrorString(_e), __FILE__, __LINE__); \
    return; } } while (0)

typedef short  s16x8 __attribute__((ext_vector_type(8)));
typedef short  s16x4 __attribute__((ext_vector_type(4)));
typedef float  f32x4 __attribute__((ext_vector_type(4)));
typedef float  f32x2 __attribute__((ext_vector_type(2)));
typedef __bf16 bf16x2 __attribute__((ext_vector_type(2)));
typedef unsigned u32x4 __attribute__((ext_vector_type(4)));
typedef unsigned u32x2 __attribute__((ext_vector_type(2)));
typedef __attribute__((address_space(3))) char lds_char;

constexpr int NR = 8;
constexpr int MS = 512, KS = 512, KT = 4096, NT = 16384;
constexpr int STAGE_B = 49152, A_B = 32768;
constexpr int LDS_B = 3 * STAGE_B;
constexpr size_t WS_CNT = 0;
constexpr size_t WS_TMO = 4096;
constexpr size_t WS_FLG = 8192;
constexpr size_t WS_STG = 1u << 20;
constexpr size_t SLOT_B = (size_t)NR * MS * KS * 2;
constexpr unsigned NSLOT = 8;
constexpr unsigned long long DEADLINE_TICKS = 40000000ull;

#define WAITV(n) asm volatile("s_waitcnt vmcnt(" #n ")" ::: "memory")
#define WAITL0() asm volatile("s_waitcnt lgkmcnt(0)" ::: "memory")
#define BAR() asm volatile("s_barrier" ::: "memory")
#define SCHED() __builtin_amdgcn_sched_barrier(0)

__device__ __forceinline__ void glds(unsigned voff, const char* sbase, unsigned ldsaddr) {
  asm volatile("s_mov_b32 m0, %2\n\ts_nop 0\n\tglobal_load_lds_dwordx4 %0, %1" :: "v"(voff), "s"(sbase), "s"(ldsaddr) : "memory");
}
__device__ __forceinline__ void glds128(unsigned voff, const char* sbase, unsigned ldsaddr) {
  asm volatile("s_mov_b32 m0, %2\n\ts_nop 0\n\tglobal_load_lds_dwordx4 %0, %1 offset:128" :: "v"(voff), "s"(sbase), "s"(ldsaddr - 128u) : "memory");
}

__device__ __forceinline__ float silu_f(float y) {
  float c = __builtin_amdgcn_fmed3f(y, -60.f, 60.f);
  float e = __builtin_amdgcn_exp2f(c * -1.4426950408889634f);
  return y * __builtin_amdgcn_rcpf(1.f + e);
}

__global__ __launch_bounds__(512, 2)
void a2a_gemm_silu(const char* __restrict__ x, const char* __restrict__ w, char* __restrict__ out,
                   char* const* __restrict__ ws_all, int rank, int mode) {
  extern __shared__ __attribute__((aligned(1024))) char smem[];
  lds_char* lds = (lds_char*)smem;
  const unsigned lds0 = (unsigned)(size_t)lds;
  const int tid = threadIdx.x, lane = tid & 63;
  const int wv = __builtin_amdgcn_readfirstlane(tid >> 6);
  const int g = wv >> 2, wg = wv & 3, wm = wg >> 1, wn = wg & 1;
  const int bid = blockIdx.x;
  const int xcd = bid & 7, jj = bid >> 3, mh = jj & 1, ntile = xcd * 16 + (jj >> 1);
  const bool comm = (mode & 1) != 0;
  char* const ws = ws_all[rank];

  const int myround = (comm && bid < 224) ? bid / 28 : -100;
  const int prem = bid % 28, pdest = (rank + 8 - (prem / 4 + 1)) & 7, pq = prem & 3;

  unsigned cntv; u32x4 px0, px1;
  const unsigned voffP = (unsigned)((tid >> 3) * 1024 + (tid & 7) * 16);
  {
    const char* cp = ws + WS_CNT + bid * 4;
    asm volatile("global_load_dword %0, %1, off sc0 sc1" : "=v"(cntv) : "v"(cp) : "memory");
    const int rr = (myround >= 0) ? myround : 0;
    const char* xp = x + (size_t)(pdest * MS + pq * 128) * 1024 + rr * 128;
    asm volatile("global_load_dwordx4 %0, %1, %2" : "=v"(px0) : "v"(voffP), "s"(xp) : "memory");
    asm volatile("global_load_dwordx4 %0, %1, %2" : "=v"(px1) : "v"(voffP), "s"(xp + 65536) : "memory");
  }

  const int r8 = lane >> 3;
  const unsigned voffA0 = (unsigned)((64 * wg + r8) * 1024 + (((lane & 7) ^ (r8 >> 1)) * 16));
  const unsigned voffA1 = (voffA0 ^ 64u) + 8192u;
  const int rb = (lane >> 2) & 7, sub = lane >> 5, slot4 = lane & 3;
  const unsigned voffB0 = (unsigned)((16 * wg + rb) * 32768 + sub * 64 + ((slot4 ^ (rb >> 2)) * 16));
  const unsigned voffB1 = (unsigned)((16 * wg + 8 + rb) * 32768 + sub * 64 + ((slot4 ^ (2 | (rb >> 2))) * 16));
  const char* const xloc = x + (size_t)(rank * MS + mh * 256) * 1024;
  const char* const wcol = w + (size_t)ntile * 256;

  auto stage_tile = [&](const char* abase, const char* bbase, unsigned sbyte) {
    const unsigned la = lds0 + sbyte + wg * 8192;
#pragma unroll
    for (int j = 0; j < 4; ++j) {
      glds(voffA0, abase + j * 16384, la + j * 2048);
      glds(voffA1, abase + j * 16384, la + j * 2048 + 1024);
    }
    const unsigned lb = lds0 + sbyte + A_B + wg * 4096;
    glds(voffB0, bbase, lb);
    glds128(voffB0, bbase, lb + 1024);
    glds(voffB1, bbase, lb + 2048);
    glds128(voffB1, bbase, lb + 3072);
  };

  unsigned pollv = 0;
  stage_tile(xloc + g * 128, wcol + (size_t)(rank * KS + g * 64) * 32768, g * STAGE_B);
  asm volatile("s_waitcnt vmcnt(12)" : "+v"(cntv), "+v"(px0), "+v"(px1) :: "memory");
  const unsigned call = __builtin_amdgcn_readfirstlane(cntv) + 1u;
  const size_t slot_off = WS_STG + (size_t)(call % NSLOT) * SLOT_B;
  const char* const stg = ws + slot_off + (size_t)mh * 256 * 1024;
  const char* const flg = ws + WS_FLG + mh * 128;
  unsigned* const tmo = (unsigned*)(ws + WS_TMO);

  char* pdst = nullptr; char* pflag = nullptr;
  if (myround >= 0) {
    char* wsd = ws_all[(unsigned)pdest < (unsigned)NR ? pdest : rank];
    pdst = wsd + slot_off + (size_t)rank * (MS * KS * 2) + (size_t)(pq * 128) * 1024 + myround * 128;
    pflag = wsd + WS_FLG + ((rank * 8 + myround) * 2 + (pq >> 1)) * 128 + ((pq & 1) * 2 + g) * 4;
  }
  auto push = [&]() {
    asm volatile("global_store_dwordx4 %0, %1, %2 sc0 sc1\n\ts_nop 1" :: "v"(voffP), "v"(px0), "s"(pdst) : "memory");
    asm volatile("global_store_dwordx4 %0, %1, %2 sc0 sc1\n\ts_nop 1" :: "v"(voffP), "v"(px1), "s"(pdst + 65536) : "memory");
  };
  if (myround == 0) { push(); WAITV(2); } else { WAITV(0); }
  BAR();
  if (g == 1) BAR();

  const int fr = lane & 15, fq = lane >> 4;
  unsigned rdA = lds0 + g * STAGE_B + (unsigned)((128 * wm + fr) * 128 + ((fq ^ (fr >> 1)) * 16));
  unsigned rdB = lds0 + g * STAGE_B + (unsigned)(A_B + 2048 * fq + 1024 * wn + 64 * ((lane & 15) >> 2) + 8 * (lane & 1)
                                                 + 16 * (((lane >> 1) & 1) ^ ((fq & 1) << 1)));

  f32x4 acc[8][4];
#pragma unroll
  for (int m = 0; m < 8; ++m)
#pragma unroll
    for (int n = 0; n < 4; ++n) acc[m][n] = f32x4{0.f, 0.f, 0.f, 0.f};

  s16x8 fa[8][2], fb[4][2];
#pragma unroll
  for (int n = 0; n < 4; ++n) fb[n][1] = s16x8{0, 0, 0, 0, 0, 0, 0, 0};
  fa[6][1] = fb[0][1]; fa[7][1] = fb[0][1];
  unsigned sread = g * STAGE_B;
  unsigned swrite = (g == 0) ? 2 * STAGE_B : 0;
  const bool leader = (wg == 0);
  const char* abase = xloc + (g + 2) * 128;
  const char* bbase = wcol + (size_t)(rank * KS + (g + 2) * 64) * 32768;
  const char* const xl2 = xloc - (size_t)rank * (MS * KS * 2);
  const unsigned long long dstg = (unsigned long long)(stg - xl2);
  int pj = g - 4, prd = 0;

#define MF(m, n, ks) acc[m][n] = __builtin_amdgcn_mfma_f32_16x16x32_bf16(fb[n][ks], fa[m][ks], acc[m][n], 0, 0, 0)
#define RB(n, ks) { lds_char* pb0 = pbb[(n) & 1][0] + (8192 * (ks) + 512 * ((n) >> 1)); lds_char* pb1 = pbb[(n) & 1][1] + (8192 * (ks) + 512 * ((n) >> 1) + 256); \
    s16x4 lo = __builtin_amdgcn_ds_read_tr16_b64_v4i16((__attribute__((address_space(3))) s16x4*)pb0); \
    s16x4 hi = __builtin_amdgcn_ds_read_tr16_b64_v4i16((__attribute__((address_space(3))) s16x4*)pb1); \
    fb[n][ks] = __builtin_shufflevector(lo, hi, 0, 1, 2, 3, 4, 5, 6, 7); }
#define RA0(m) fa[m][0] = *(__attribute__((address_space(3))) s16x8*)(pa0 + 2048 * (m))
#define RA1(m) fa[m][1] = *(__attribute__((address_space(3))) s16x8*)(pa1 + 2048 * (m))

  for (int i = 0; i < 32; ++i) {
    __builtin_amdgcn_s_setprio(3);
    {
      asm volatile("" : "+v"(rdA), "+v"(rdB));
      lds_char* pa0 = (lds_char*)(size_t)rdA;
      lds_char* pa1 = (lds_char*)(size_t)(rdA ^ 64u);
      lds_char* pbb[2][2] = {{(lds_char*)(size_t)rdB, (lds_char*)(size_t)(rdB ^ 16u)},
                             {(lds_char*)(size_t)(rdB ^ 32u), (lds_char*)(size_t)(rdB ^ 48u)}};
      const unsigned la = lds0 + swrite + wg * 8192;
      const unsigned lb = lds0 + swrite + A_B + wg * 4096;
      SCHED();
      glds(voffB0, bbase, lb);
      MF(6, 0, 1); MF(6, 1, 1); RB(0, 0); RB(1, 0); SCHED();
      glds128(voffB0, bbase, lb + 1024);
      MF(6, 2, 1); MF(6, 3, 1); RB(2, 0); RB(3, 0); SCHED();
      glds(voffB1, bbase, lb + 2048);
      MF(7, 0, 1); MF(7, 1, 1); RA0(0); RA0(1); SCHED();
      glds128(voffB1, bbase, lb + 3072);
      MF(7, 2, 1); MF(7, 3, 1); RA0(2); RA0(3); SCHED();
      glds(voffA0, abase, la);
      RA0(4); RA0(5); RA0(6); SCHED();
      glds(voffA1, abase, la + 1024);
      RA0(7); RB(0, 1); RB(1, 1); SCHED();
      glds(voffA0, abase + 16384, la + 2048);
      RB(2, 1); RB(3, 1); RA1(0); SCHED();
      glds(voffA1, abase + 16384, la + 3072);
      RA1(1); RA1(2); RA1(3); SCHED();
      glds(voffA0, abase + 32768, la + 4096);
      RA1(4); RA1(5); SCHED();
      glds(voffA1, abase + 32768, la + 5120);
      RA1(6); RA1(7); SCHED();
      glds(voffA0, abase + 49152, la + 6144);
      glds(voffA1, abase + 49152, la + 7168);
      SCHED();
    }
    const int psrc = (rank + pj + 1) & 7;
    const bool prem = pj >= 0 && prd < 8;
    const bool dopoll = comm && leader && prem;
    const char* pollp = flg + (psrc * 8 + prd) * 256;
    if (dopoll)
      asm volatile("v_and_b32 %0, 48, %1\n\tv_lshrrev_b32 %0, 2, %0\n\tglobal_load_dword %0, %0, %2 sc0 sc1" : "=&v"(pollv) : "v"(voffP), "s"(pollp) : "memory");
    int pend = 0;
    if (i == myround - 1) { push(); pend = 2; }
    if (i == myround + 1 && leader) {
      unsigned t0v, t1v;
      asm volatile("v_mov_b32 %0, 0\n\tv_mov_b32 %1, %3\n\tglobal_store_dword %0, %1, %2 sc0 sc1\n\ts_nop 1" : "=&v"(t0v), "=&v"(t1v) : "s"(pflag), "s"(call) : "memory");
      pend = 1;
    }
    WAITL0();
    __builtin_amdgcn_s_setprio(0);
    BAR();
    SCHED();
#pragma unroll
    for (int m = 0; m < 2; ++m)
#pragma unroll
      for (int n = 0; n < 4; ++n) MF(m, n, 0);
    SCHED();
    {
      const unsigned mk = (unsigned)(~pj >> 31) & (unsigned)((prd - 8) >> 31);
      const unsigned lrd = (unsigned)(pj + 8) & (unsigned)(pj >> 31);
      const unsigned kt = ((unsigned)(psrc * 8 + prd) & mk) | (((unsigned)rank * 8u + lrd) & ~mk);
      abase = xl2 + (dstg & (unsigned long long)(long long)(int)mk) + (size_t)(((kt >> 3) << 19) | ((kt & 7u) << 7));
      bbase = wcol + (size_t)(kt << 21);
      pj += 2;
      const int wrap = (6 - pj) >> 31;
      pj += 7 * wrap; prd -= wrap;
      const unsigned nx = (swrite >= STAGE_B) ? swrite - STAGE_B : swrite + 2 * STAGE_B;
      rdA += swrite - sread; rdB += swrite - sread;
      sread = swrite; swrite = nx;
    }
#pragma unroll
    for (int m = 2; m < 8; ++m)
#pragma unroll
      for (int n = 0; n < 4; ++n) MF(m, n, 0);
#pragma unroll
    for (int k = 0; k < 24; ++k) { __builtin_amdgcn_sched_group_barrier(0x008, 1, 0); __builtin_amdgcn_sched_group_barrier(0x006, 2, 0); }
    SCHED();
#pragma unroll
    for (int m = 0; m < 6; ++m)
#pragma unroll
      for (int n = 0; n < 4; ++n) MF(m, n, 1);
    SCHED();
    if (pend == 0) WAITV(0); else if (pend == 1) WAITV(1); else WAITV(2);
    if (dopoll) {
      asm volatile("" : "+v"(pollv));
      if (__builtin_amdgcn_ballot_w64(pollv < call) != 0ull) {
        unsigned long long t0 = __builtin_amdgcn_s_memrealtime();
        for (;;) {
          unsigned a = __hip_atomic_load((const unsigned*)pollp + (lane & 3), __ATOMIC_RELAXED, __HIP_MEMORY_SCOPE_SYSTEM);
          if (__builtin_amdgcn_ballot_w64(a < call) == 0ull) break;
          if (__hip_atomic_load(tmo, __ATOMIC_RELAXED, __HIP_MEMORY_SCOPE_AGENT) == call ||
              __builtin_amdgcn_s_memrealtime() - t0 > DEADLINE_TICKS) {
            __hip_atomic_store(tmo, call, __ATOMIC_RELAXED, __HIP_MEMORY_SCOPE_AGENT);
            break;
          }
          __builtin_amdgcn_s_sleep(1);
        }
        WAITV(0);
      }
    }
    BAR();
    SCHED();
  }
  __builtin_amdgcn_s_setprio(3);
#pragma unroll
  for (int m = 6; m < 8; ++m)
#pragma unroll
    for (int n = 0; n < 4; ++n) MF(m, n, 1);
  __builtin_amdgcn_s_setprio(0);
  SCHED();
  if (g == 0) BAR();

  WAITV(0);
  const bool bad = comm && (__hip_atomic_load(tmo, __ATOMIC_RELAXED, __HIP_MEMORY_SCOPE_AGENT) == call);
  {
    lds_char* mine = lds + wv * 16384 + lane * 16;
#pragma unroll
    for (int m = 0; m < 4; ++m)
#pragma unroll
      for (int n = 0; n < 4; ++n) {
        f32x4 v = (g == 0) ? acc[m + 4][n] : acc[m][n];
        *(__attribute__((address_space(3))) f32x4*)(mine + (m * 4 + n) * 1024) = v;
      }
  }
  WAITL0();
  BAR();
  lds_char* theirs = lds + (wv ^ 4) * 16384;
  f32x4 fin[4][4];
#pragma unroll
  for (int m = 0; m < 4; ++m)
#pragma unroll
    for (int n = 0; n < 4; ++n) {
      f32x4 o = *(__attribute__((address_space(3))) f32x4*)(theirs + lane * 16 + (m * 4 + n) * 1024);
      f32x4 v = (g == 0) ? acc[m][n] : acc[m + 4][n];
      fin[m][n] = v + o;
    }
  WAITL0();
  SCHED();
#pragma unroll
  for (int m = 0; m < 4; ++m)
#pragma unroll
    for (int n = 0; n < 4; ++n) {
      f32x4 v = fin[m][n];
      bf16x2 lo = __builtin_convertvector(f32x2{silu_f(v[0]), silu_f(v[1])}, bf16x2);
      bf16x2 hi = __builtin_convertvector(f32x2{silu_f(v[2]), silu_f(v[3])}, bf16x2);
      u32x2 pk = u32x2{__builtin_bit_cast(unsigned, lo), __builtin_bit_cast(unsigned, hi)};
      if (bad) pk = u32x2{0x7fc07fc0u, 0x7fc07fc0u};
      const int row = 16 * m + fr, c = 2 * n + (fq >> 1);
      *(__attribute__((address_space(3))) u32x2*)(theirs + row * 128 + ((c ^ (row & 7)) * 16) + (fq & 1) * 8) = pk;
    }
  WAITL0();
  SCHED();
  {
    char* op = out + (size_t)(mh * 256 + wm * 128 + g * 64 + r8) * 32768 + (size_t)ntile * 256 + wn * 128 + (lane & 7) * 16;
#pragma unroll
    for (int i = 0; i < 8; ++i) {
      const int row = i * 8 + r8;
      u32x4 v = *(__attribute__((address_space(3))) u32x4*)(theirs + row * 128 + (((lane & 7) ^ (row & 7)) * 16));
      *(u32x4*)(op + (size_t)i * 8 * 32768) = v;
    }
  }
  if (tid == 0) __hip_atomic_store((unsigned*)(ws + WS_CNT + bid * 4), call, __ATOMIC_RELAXED, __HIP_MEMORY_SCOPE_AGENT);
}

extern "C" void kernel_launch(int rank, int nranks, ncclComm_t comm,
                              hipStream_t stream,
                              void* const* d_in, const int* in_sizes, int n_in,
                              void* d_out, int out_size,
                              const int* in_dtypes, int out_dtype,
                              void* const* d_out_all,
                              void* const* d_ws_all, size_t ws_bytes) {
  if (nranks != NR || ws_bytes < WS_STG + NSLOT * SLOT_B) { fprintf(stderr, "unexpected configuration\n"); return; }
  static bool init = false;
  if (!init) {
    HIP_CHECK(hipFuncSetAttribute((const void*)a2a_gemm_silu, hipFuncAttributeMaxDynamicSharedMemorySize, LDS_B));
    init = true;
  }
  hipLaunchKernelGGL(a2a_gemm_silu, dim3(256), dim3(512), LDS_B, stream,
                     (const char*)d_in[0], (const char*)d_in[1], (char*)d_out,
                     (char* const*)d_ws_all, rank, 1);
  HIP_CHECK(hipGetLastError());
}
```

```cpp
#define BPOL "nt"
#include <hip/hip_runtime.h>
#include <rccl/rccl.h>
#include <cstdio>

#define HIP_CHECK(expr) do { hipError_t _e = (expr); if (_e != hipSuccess) { \
    fprintf(stderr, "HIP error %s at %s:%d\n", hipGetErrorString(_e), __FILE__, __LINE__); \
    return; } } while (0)

typedef short  s16x8 __attribute__((ext_vector_type(8)));
typedef short  s16x4 __attribute__((ext_vector_type(4)));
typedef float  f32x4 __attribute__((ext_vector_type(4)));
typedef float  f32x2 __attribute__((ext_vector_type(2)));
typedef __bf16 bf16x2 __attribute__((ext_vector_type(2)));
typedef unsigned u32x4 __attribute__((ext_vector_type(4)));
typedef unsigned u32x2 __attribute__((ext_vector_type(2)));
typedef __attribute__((address_space(3))) char lds_char;

constexpr int NR = 8;
constexpr int MS = 512, KS = 512, KT = 4096, NT = 16384;
constexpr int STAGE_B = 49152, A_B = 32768;
constexpr int LDS_B = 3 * STAGE_B;
constexpr size_t WS_CNT = 0;
constexpr size_t WS_TMO = 4096;
constexpr size_t WS_FLG = 8192;
constexpr size_t WS_STG = 1u << 20;
constexpr size_t SLOT_B = (size_t)NR * MS * KS * 2;
constexpr unsigned NSLOT = 8;
constexpr unsigned long long DEADLINE_TICKS = 40000000ull;

#define WAITV(n) asm volatile("s_waitcnt vmcnt(" #n ")" ::: "memory")
#define WAITL0() asm volatile("s_waitcnt lgkmcnt(0)" ::: "memory")
#define BAR() asm volatile("s_barrier" ::: "memory")
#define SCHED() __builtin_amdgcn_sched_barrier(0)

__device__ __forceinline__ void glds(unsigned voff, const char* sbase, unsigned ldsaddr) {
  asm volatile("s_mov_b32 m0, %2\n\ts_nop 0\n\tglobal_load_lds_dwordx4 %0, %1" :: "v"(voff), "s"(sbase), "s"(ldsaddr) : "memory");
}
__device__ __forceinline__ void gldsb(unsigned voff, const char* sbase, unsigned ldsaddr) {
  asm volatile("s_mov_b32 m0, %2\n\ts_nop 0\n\tglobal_load_lds_dwordx4 %0, %1 " BPOL :: "v"(voff), "s"(sbase), "s"(ldsaddr) : "memory");
}
__device__ __forceinline__ void glds128(unsigned voff, const char* sbase, unsigned ldsaddr) {
  asm volatile("s_mov_b32 m0, %2\n\ts_nop 0\n\tglobal_load_lds_dwordx4 %0, %1 offset:128 " BPOL :: "v"(voff), "s"(sbase), "s"(ldsaddr - 128u) : "memory");
}

__device__ __forceinline__ float silu_f(float y) {
  float c = __builtin_amdgcn_fmed3f(y, -60.f, 60.f);
  float e = __builtin_amdgcn_exp2f(c * -1.4426950408889634f);
  return y * __builtin_amdgcn_rcpf(1.f + e);
}

__global__ __launch_bounds__(512, 2)
void a2a_gemm_silu(const char* __restrict__ x, const char* __restrict__ w, char* __restrict__ out,
                   char* const* __restrict__ ws_all, int rank, int mode) {
  extern __shared__ __attribute__((aligned(1024))) char smem[];
  lds_char* lds = (lds_char*)smem;
  const unsigned lds0 = (unsigned)(size_t)lds;
  const int tid = threadIdx.x, lane = tid & 63;
  const int wv = __builtin_amdgcn_readfirstlane(tid >> 6);
  const int g = wv >> 2, wg = wv & 3, wm = wg >> 1, wn = wg & 1;
  const int bid = blockIdx.x;
  const int xcd = bid & 7, jj = bid >> 3, mh = jj & 1, ntile = xcd * 16 + (jj >> 1);
  const bool comm = (mode & 1) != 0;
  char* const ws = ws_all[rank];

  const int myround = (comm && bid < 224) ? bid / 28 : -100;
  const int prem = bid % 28, pdest = (rank + 8 - (prem / 4 + 1)) & 7, pq = prem & 3;

  unsigned cntv; u32x4 px0, px1;
  const unsigned voffP = (unsigned)((tid >> 3) * 1024 + (tid & 7) * 16);
  {
    const char* cp = ws + WS_CNT + bid * 4;
    asm volatile("global_load_dword %0, %1, off sc0 sc1" : "=v"(cntv) : "v"(cp) : "memory");
    const int rr = (myround >= 0) ? myround : 0;
    const char* xp = x + (size_t)(pdest * MS + pq * 128) * 1024 + rr * 128;
    asm volatile("global_load_dwordx4 %0, %1, %2" : "=v"(px0) : "v"(voffP), "s"(xp) : "memory");
    asm volatile("global_load_dwordx4 %0, %1, %2" : "=v"(px1) : "v"(voffP), "s"(xp + 65536) : "memory");
  }

  const int r8 = lane >> 3;
  const unsigned voffA0 = (unsigned)((64 * wg + r8) * 1024 + (((lane & 7) ^ (r8 >> 1)) * 16));
  const unsigned voffA1 = (voffA0 ^ 64u) + 8192u;
  const int rb = (lane >> 2) & 7, sub = lane >> 5, slot4 = lane & 3;
  const unsigned voffB0 = (unsigned)((16 * wg + rb) * 32768 + sub * 64 + ((slot4 ^ (rb >> 2)) * 16));
  const unsigned voffB1 = (unsigned)((16 * wg + 8 + rb) * 32768 + sub * 64 + ((slot4 ^ (2 | (rb >> 2))) * 16));
  const char* const xloc = x + (size_t)(rank * MS + mh * 256) * 1024;
  const char* const wcol = w + (size_t)ntile * 256;

  auto stage_tile = [&](const char* abase, const char* bbase, unsigned sbyte) {
    const unsigned la = lds0 + sbyte + wg * 8192;
#pragma unroll
    for (int j = 0; j < 4; ++j) {
      glds(voffA0, abase + j * 16384, la + j * 2048);
      glds(voffA1, abase + j * 16384, la + j * 2048 + 1024);
    }
    const unsigned lb = lds0 + sbyte + A_B + wg * 4096;
    gldsb(voffB0, bbase, lb);
    glds128(voffB0, bbase, lb + 1024);
    gldsb(voffB1, bbase, lb + 2048);
    glds128(voffB1, bbase, lb + 3072);
  };

  unsigned pollv = 0;
  stage_tile(xloc + g * 128, wcol + (size_t)(rank * KS + g * 64) * 32768, g * STAGE_B);
  asm volatile("s_waitcnt vmcnt(12)" : "+v"(cntv), "+v"(px0), "+v"(px1) :: "memory");
  const unsigned call = __builtin_amdgcn_readfirstlane(cntv) + 1u;
  const size_t slot_off = WS_STG + (size_t)(call % NSLOT) * SLOT_B;
  const char* const stg = ws + slot_off + (size_t)mh * 256 * 1024;
  const char* const flg = ws + WS_FLG + mh * 128;
  unsigned* const tmo = (unsigned*)(ws + WS_TMO);

  char* pdst = nullptr; char* pflag = nullptr;
  if (myround >= 0) {
    char* wsd = ws_all[(unsigned)pdest < (unsigned)NR ? pdest : rank];
    pdst = wsd + slot_off + (size_t)rank * (MS * KS * 2) + (size_t)(pq * 128) * 1024 + myround * 128;
    pflag = wsd + WS_FLG + ((rank * 8 + myround) * 2 + (pq >> 1)) * 128 + ((pq & 1) * 2 + g) * 4;
  }
  auto push = [&]() {
    asm volatile("global_store_dwordx4 %0, %1, %2 sc0 sc1\n\ts_nop 1" :: "v"(voffP), "v"(px0), "s"(pdst) : "memory");
    asm volatile("global_store_dwordx4 %0, %1, %2 sc0 sc1\n\ts_nop 1" :: "v"(voffP), "v"(px1), "s"(pdst + 65536) : "memory");
  };
  if (myround == 0) { push(); WAITV(2); } else { WAITV(0); }
  BAR();
  if (g == 1) BAR();

  const int fr = lane & 15, fq = lane >> 4;
  unsigned rdA = lds0 + g * STAGE_B + (unsigned)((128 * wm + fr) * 128 + ((fq ^ (fr >> 1)) * 16));
  unsigned rdB = lds0 + g * STAGE_B + (unsigned)(A_B + 2048 * fq + 1024 * wn + 64 * ((lane & 15) >> 2) + 8 * (lane & 1)
                                                 + 16 * (((lane >> 1) & 1) ^ ((fq & 1) << 1)));

  f32x4 acc[8][4];
#pragma unroll
  for (int m = 0; m < 8; ++m)
#pragma unroll
    for (int n = 0; n < 4; ++n) acc[m][n] = f32x4{0.f, 0.f, 0.f, 0.f};

  s16x8 fa[8][2], fb[4][2];
#pragma unroll
  for (int n = 0; n < 4; ++n) fb[n][1] = s16x8{0, 0, 0, 0, 0, 0, 0, 0};
  fa[6][1] = fb[0][1]; fa[7][1] = fb[0][1];
  unsigned sread = g * STAGE_B;
  unsigned swrite = (g == 0) ? 2 * STAGE_B : 0;
  const bool leader = (wg == 0);
  const char* abase = xloc + (g + 2) * 128;
  const char* bbase = wcol + (size_t)(rank * KS + (g + 2) * 64) * 32768;
  const char* const xl2 = xloc - (size_t)rank * (MS * KS * 2);
  const unsigned long long dstg = (unsigned long long)(stg - xl2);
  int pj = g - 4, prd = 0;

#define MF(m, n, ks) acc[m][n] = __builtin_amdgcn_mfma_f32_16x16x32_bf16(fb[n][ks], fa[m][ks], acc[m][n], 0, 0, 0)
#define RB(n, ks) { lds_char* pb0 = pbb[(n) & 1][0] + (8192 * (ks) + 512 * ((n) >> 1)); lds_char* pb1 = pbb[(n) & 1][1] + (8192 * (ks) + 512 * ((n) >> 1) + 256); \
    s16x4 lo = __builtin_amdgcn_ds_read_tr16_b64_v4i16((__attribute__((address_space(3))) s16x4*)pb0); \
    s16x4 hi = __builtin_amdgcn_ds_read_tr16_b64_v4i16((__attribute__((address_space(3))) s16x4*)pb1); \
    fb[n][ks] = __builtin_shufflevector(lo, hi, 0, 1, 2, 3, 4, 5, 6, 7); }
#define RA0(m) fa[m][0] = *(__attribute__((address_space(3))) s16x8*)(pa0 + 2048 * (m))
#define RA1(m) fa[m][1] = *(__attribute__((address_space(3))) s16x8*)(pa1 + 2048 * (m))

  for (int i = 0; i < 32; ++i) {
    __builtin_amdgcn_s_setprio(3);
    {
      asm volatile("" : "+v"(rdA), "+v"(rdB));
      lds_char* pa0 = (lds_char*)(size_t)rdA;
      lds_char* pa1 = (lds_char*)(size_t)(rdA ^ 64u);
      lds_char* pbb[2][2] = {{(lds_char*)(size_t)rdB, (lds_char*)(size_t)(rdB ^ 16u)},
                             {(lds_char*)(size_t)(rdB ^ 32u), (lds_char*)(size_t)(rdB ^ 48u)}};
      const unsigned la = lds0 + swrite + wg * 8192;
      const unsigned lb = lds0 + swrite + A_B + wg * 4096;
      SCHED();
      gldsb(voffB0, bbase, lb);
      MF(6, 0, 1); MF(6, 1, 1); RB(0, 0); RB(1, 0); SCHED();
      glds128(voffB0, bbase, lb + 1024);
      MF(6, 2, 1); MF(6, 3, 1); RB(2, 0); RB(3, 0); SCHED();
      gldsb(voffB1, bbase, lb + 2048);
      MF(7, 0, 1); MF(7, 1, 1); RA0(0); RA0(1); SCHED();
      glds128(voffB1, bbase, lb + 3072);
      MF(7, 2, 1); MF(7, 3, 1); RA0(2); RA0(3); SCHED();
      glds(voffA0, abase, la);
      RA0(4); RA0(5); RA0(6); SCHED();
      glds(voffA1, abase, la + 1024);
      RA0(7); RB(0, 1); RB(1, 1); SCHED();
      glds(voffA0, abase + 16384, la + 2048);
      RB(2, 1); RB(3, 1); RA1(0); SCHED();
      glds(voffA1, abase + 16384, la + 3072);
      RA1(1); RA1(2); RA1(3); SCHED();
      glds(voffA0, abase + 32768, la + 4096);
      RA1(4); RA1(5); SCHED();
      glds(voffA1, abase + 32768, la + 5120);
      RA1(6); RA1(7); SCHED();
      glds(voffA0, abase + 49152, la + 6144);
      glds(voffA1, abase + 49152, la + 7168);
      SCHED();
    }
    const int psrc = (rank + pj + 1) & 7;
    const bool prem = pj >= 0 && prd < 8;
    const bool dopoll = comm && leader && prem;
    const char* pollp = flg + (psrc * 8 + prd) * 256;
    if (dopoll)
      asm volatile("v_and_b32 %0, 48, %1\n\tv_lshrrev_b32 %0, 2, %0\n\tglobal_load_dword %0, %0, %2 sc0 sc1" : "=&v"(pollv) : "v"(voffP), "s"(pollp) : "memory");
    int pend = 0;
    if (i == myround - 1) { push(); pend = 2; }
    if (i == myround + 1 && leader) {
      unsigned t0v, t1v;
      asm volatile("v_mov_b32 %0, 0\n\tv_mov_b32 %1, %3\n\tglobal_store_dword %0, %1, %2 sc0 sc1\n\ts_nop 1" : "=&v"(t0v), "=&v"(t1v) : "s"(pflag), "s"(call) : "memory");
      pend = 1;
    }
    WAITL0();
    __builtin_amdgcn_s_setprio(0);
    BAR();
    SCHED();
#pragma unroll
    for (int m = 0; m < 2; ++m)
#pragma unroll
      for (int n = 0; n < 4; ++n) MF(m, n, 0);
    SCHED();
    {
      const unsigned mk = (unsigned)(~pj >> 31) & (unsigned)((prd - 8) >> 31);
      const unsigned lrd = (unsigned)(pj + 8) & (unsigned)(pj >> 31);
      const unsigned kt = ((unsigned)(psrc * 8 + prd) & mk) | (((unsigned)rank * 8u + lrd) & ~mk);
      abase = xl2 + (dstg & (unsigned long long)(long long)(int)mk) + (size_t)(((kt >> 3) << 19) | ((kt & 7u) << 7));
      bbase = wcol + (size_t)(kt << 21);
      pj += 2;
      const int wrap = (6 - pj) >> 31;
      pj += 7 * wrap; prd -= wrap;
      const unsigned nx = (swrite >= STAGE_B) ? swrite - STAGE_B : swrite + 2 * STAGE_B;
      rdA += swrite - sread; rdB += swrite - sread;
      sread = swrite; swrite = nx;
    }
#pragma unroll
    for (int m = 2; m < 8; ++m)
#pragma unroll
      for (int n = 0; n < 4; ++n) MF(m, n, 0);
#pragma unroll
    for (int k = 0; k < 24; ++k) { __builtin_amdgcn_sched_group_barrier(0x008, 1, 0); __builtin_amdgcn_sched_group_barrier(0x006, 2, 0); }
    SCHED();
#pragma unroll
    for (int m = 0; m < 6; ++m)
#pragma unroll
      for (int n = 0; n < 4; ++n) MF(m, n, 1);
    SCHED();
    if (pend == 0) WAITV(0); else if (pend == 1) WAITV(1); else WAITV(2);
    if (dopoll) {
      asm volatile("" : "+v"(pollv));
      if (__builtin_amdgcn_ballot_w64(pollv < call) != 0ull) {
        unsigned long long t0 = __builtin_amdgcn_s_memrealtime();
        for (;;) {
          unsigned a = __hip_atomic_load((const unsigned*)pollp + (lane & 3), __ATOMIC_RELAXED, __HIP_MEMORY_SCOPE_SYSTEM);
          if (__builtin_amdgcn_ballot_w64(a < call) == 0ull) break;
          if (__hip_atomic_load(tmo, __ATOMIC_RELAXED, __HIP_MEMORY_SCOPE_AGENT) == call ||
              __builtin_amdgcn_s_memrealtime() - t0 > DEADLINE_TICKS) {
            __hip_atomic_store(tmo, call, __ATOMIC_RELAXED, __HIP_MEMORY_SCOPE_AGENT);
            break;
          }
          __builtin_amdgcn_s_sleep(1);
        }
        WAITV(0);
      }
    }
    BAR();
    SCHED();
  }
  __builtin_amdgcn_s_setprio(3);
#pragma unroll
  for (int m = 6; m < 8; ++m)
#pragma unroll
    for (int n = 0; n < 4; ++n) MF(m, n, 1);
  __builtin_amdgcn_s_setprio(0);
  SCHED();
  if (g == 0) BAR();

  WAITV(0);
  const bool bad = comm && (__hip_atomic_load(tmo, __ATOMIC_RELAXED, __HIP_MEMORY_SCOPE_AGENT) == call);
  {
    lds_char* mine = lds + wv * 16384 + lane * 16;
#pragma unroll
    for (int m = 0; m < 4; ++m)
#pragma unroll
      for (int n = 0; n < 4; ++n) {
        f32x4 v = (g == 0) ? acc[m + 4][n] : acc[m][n];
        *(__attribute__((address_space(3))) f32x4*)(mine + (m * 4 + n) * 1024) = v;
      }
  }
  WAITL0();
  BAR();
  lds_char* theirs = lds + (wv ^ 4) * 16384;
  f32x4 fin[4][4];
#pragma unroll
  for (int m = 0; m < 4; ++m)
#pragma unroll
    for (int n = 0; n < 4; ++n) {
      f32x4 o = *(__attribute__((address_space(3))) f32x4*)(theirs + lane * 16 + (m * 4 + n) * 1024);
      f32x4 v = (g == 0) ? acc[m][n] : acc[m + 4][n];
      fin[m][n] = v + o;
    }
  WAITL0();
  SCHED();
#pragma unroll
  for (int m = 0; m < 4; ++m)
#pragma unroll
    for (int n = 0; n < 4; ++n) {
      f32x4 v = fin[m][n];
      bf16x2 lo = __builtin_convertvector(f32x2{silu_f(v[0]), silu_f(v[1])}, bf16x2);
      bf16x2 hi = __builtin_convertvector(f32x2{silu_f(v[2]), silu_f(v[3])}, bf16x2);
      u32x2 pk = u32x2{__builtin_bit_cast(unsigned, lo), __builtin_bit_cast(unsigned, hi)};
      if (bad) pk = u32x2{0x7fc07fc0u, 0x7fc07fc0u};
      const int row = 16 * m + fr, c = 2 * n + (fq >> 1);
      *(__attribute__((address_space(3))) u32x2*)(theirs + row * 128 + ((c ^ (row & 7)) * 16) + (fq & 1) * 8) = pk;
    }
  WAITL0();
  SCHED();
  {
    char* op = out + (size_t)(mh * 256 + wm * 128 + g * 64 + r8) * 32768 + (size_t)ntile * 256 + wn * 128 + (lane & 7) * 16;
#pragma unroll
    for (int i = 0; i < 8; ++i) {
      const int row = i * 8 + r8;
      u32x4 v = *(__attribute__((address_space(3))) u32x4*)(theirs + row * 128 + (((lane & 7) ^ (row & 7)) * 16));
      __builtin_nontemporal_store(v, (u32x4*)(op + (size_t)i * 8 * 32768));
    }
  }
  if (tid == 0) __hip_atomic_store((unsigned*)(ws + WS_CNT + bid * 4), call, __ATOMIC_RELAXED, __HIP_MEMORY_SCOPE_AGENT);
}

extern "C" void kernel_launch(int rank, int nranks, ncclComm_t comm,
                              hipStream_t stream,
                              void* const* d_in, const int* in_sizes, int n_in,
                              void* d_out, int out_size,
                              const int* in_dtypes, int out_dtype,
                              void* const* d_out_all,
                              void* const* d_ws_all, size_t ws_bytes) {
  if (nranks != NR || ws_bytes < WS_STG + NSLOT * SLOT_B) { fprintf(stderr, "unexpected configuration\n"); return; }
  static bool init = false;
  if (!init) {
    HIP_CHECK(hipFuncSetAttribute((const void*)a2a_gemm_silu, hipFuncAttributeMaxDynamicSharedMemorySize, LDS_B));
    init = true;
  }
  hipLaunchKernelGGL(a2a_gemm_silu, dim3(256), dim3(512), LDS_B, stream,
                     (const char*)d_in[0], (const char*)d_in[1], (char*)d_out,
                     (char* const*)d_ws_all, rank, 1);
  HIP_CHECK(hipGetLastError());
}
```

```cpp
#define BPOL ""
#include <hip/hip_runtime.h>
#include <rccl/rccl.h>
#include <cstdio>

#define HIP_CHECK(expr) do { hipError_t _e = (expr); if (_e != hipSuccess) { \
    fprintf(stderr, "HIP error %s at %s:%d\n", hipGetErrorString(_e), __FILE__, __LINE__); \
    return; } } while (0)

typedef short  s16x8 __attribute__((ext_vector_type(8)));
typedef short  s16x4 __attribute__((ext_vector_type(4)));
typedef float  f32x4 __attribute__((ext_vector_type(4)));
typedef float  f32x2 __attribute__((ext_vector_type(2)));
typedef __bf16 bf16x2 __attribute__((ext_vector_type(2)));
typedef unsigned u32x4 __attribute__((ext_vector_type(4)));
typedef unsigned u32x2 __attribute__((ext_vector_type(2)));
typedef __attribute__((address_space(3))) char lds_char;

constexpr int NR = 8;
constexpr int MS = 512, KS = 512, KT = 4096, NT = 16384;
constexpr int STAGE_B = 49152, A_B = 32768;
constexpr int LDS_B = 3 * STAGE_B + 64;
constexpr size_t WS_CNT = 0;
constexpr size_t WS_TMO = 4096;
constexpr size_t WS_FLG = 8192;
constexpr size_t WS_STG = 1u << 20;
constexpr size_t SLOT_B = (size_t)NR * MS * KS * 2;
constexpr unsigned NSLOT = 8;
constexpr unsigned long long DEADLINE_TICKS = 40000000ull;

#define WAITV(n) asm volatile("s_waitcnt vmcnt(" #n ")" ::: "memory")
#define WAITL0() asm volatile("s_waitcnt lgkmcnt(0)" ::: "memory")
#define BAR() asm volatile("s_barrier" ::: "memory")
#define SCHED() __builtin_amdgcn_sched_barrier(0)

__device__ __forceinline__ void glds(unsigned voff, const char* sbase, unsigned ldsaddr) {
  asm volatile("s_mov_b32 m0, %2\n\ts_nop 0\n\tglobal_load_lds_dwordx4 %0, %1" :: "v"(voff), "s"(sbase), "s"(ldsaddr) : "memory");
}
__device__ __forceinline__ void gldsb(unsigned voff, const char* sbase, unsigned ldsaddr) {
  asm volatile("s_mov_b32 m0, %2\n\ts_nop 0\n\tglobal_load_lds_dwordx4 %0, %1 " BPOL :: "v"(voff), "s"(sbase), "s"(ldsaddr) : "memory");
}
__device__ __forceinline__ void glds128(unsigned voff, const char* sbase, unsigned ldsaddr) {
  asm volatile("s_mov_b32 m0, %2\n\ts_nop 0\n\tglobal_load_lds_dwordx4 %0, %1 offset:128 " BPOL :: "v"(voff), "s"(sbase), "s"(ldsaddr - 128u) : "memory");
}

__device__ __forceinline__ float silu_f(float y) {
  float c = __builtin_amdgcn_fmed3f(y, -60.f, 60.f);
  float e = __builtin_amdgcn_exp2f(c * -1.4426950408889634f);
  return y * __builtin_amdgcn_rcpf(1.f + e);
}

__global__ __launch_bounds__(512, 2)
void a2a_gemm_silu(const char* __restrict__ x, const char* __restrict__ w, char* __restrict__ out,
                   char* const* __restrict__ ws_all, int rank, int mode) {
  extern __shared__ __attribute__((aligned(1024))) char smem[];
  lds_char* lds = (lds_char*)smem;
  const unsigned lds0 = (unsigned)(size_t)lds;
  const int tid = threadIdx.x, lane = tid & 63;
  const int wv = __builtin_amdgcn_readfirstlane(tid >> 6);
  const int g = wv >> 2, wg = wv & 3, wm = wg >> 1, wn = wg & 1;
  const int bid = blockIdx.x;
  const int xcd = bid & 7, jj = bid >> 3, mh = jj & 1, ntile = xcd * 16 + (jj >> 1);
  const bool comm = (mode & 1) != 0;
  char* const ws = ws_all[rank];

  const int myround = (comm && bid < 224) ? bid / 28 : -100;
  const int prem = bid % 28, pdest = (rank + 8 - (prem / 4 + 1)) & 7, pq = prem & 3;

  unsigned cntv; u32x4 px0, px1;
  const unsigned voffP = (unsigned)((tid >> 3) * 1024 + (tid & 7) * 16);
  {
    const char* cp = ws + WS_CNT + bid * 4;
    asm volatile("global_load_dword %0, %1, off sc0 sc1" : "=v"(cntv) : "v"(cp) : "memory");
    const int rr = (myround >= 0) ? myround : 0;
    const char* xp = x + (size_t)(pdest * MS + pq * 128) * 1024 + rr * 128;
    asm volatile("global_load_dwordx4 %0, %1, %2" : "=v"(px0) : "v"(voffP), "s"(xp) : "memory");
    asm volatile("global_load_dwordx4 %0, %1, %2" : "=v"(px1) : "v"(voffP), "s"(xp + 65536) : "memory");
  }

  const int r8 = lane >> 3;
  const unsigned voffA0 = (unsigned)((64 * wg + r8) * 1024 + (((lane & 7) ^ (r8 >> 1)) * 16));
  const unsigned voffA1 = (voffA0 ^ 64u) + 8192u;
  const unsigned voffB0 = (unsigned)((16 * wg + r8) * 32768 + (((lane & 7) ^ (r8 & 2)) * 16));
  const unsigned voffB1 = (voffB0 ^ 64u) + 8u * 32768u;
  const char* const xloc = x + (size_t)(rank * MS + mh * 256) * 1024;
  const char* const wcol = w + (size_t)ntile * 256;

  auto stage_tile = [&](const char* abase, const char* bbase, unsigned sbyte) {
    const unsigned la = lds0 + sbyte + wg * 8192;
#pragma unroll
    for (int j = 0; j < 4; ++j) {
      glds(voffA0, abase + j * 16384, la + j * 2048);
      glds(voffA1, abase + j * 16384, la + j * 2048 + 1024);
    }
    const unsigned lb = lds0 + sbyte + A_B + wg * 4096;
    gldsb(voffB0, bbase, lb);
    glds128(voffB0, bbase, lb + 1024);
    gldsb(voffB1, bbase, lb + 2048);
    glds128(voffB1, bbase, lb + 3072);
  };

  stage_tile(xloc + g * 128, wcol + (size_t)(rank * KS + g * 64) * 32768, g * STAGE_B);
  asm volatile("s_waitcnt vmcnt(12)" : "+v"(cntv), "+v"(px0), "+v"(px1) :: "memory");
  const unsigned call = __builtin_amdgcn_readfirstlane(cntv) + 1u;
  const size_t slot_off = WS_STG + (size_t)(call % NSLOT) * SLOT_B;
  const char* const stg = ws + slot_off + (size_t)mh * 256 * 1024;
  const char* const flg = ws + WS_FLG + mh * 128;
  unsigned* const tmo = (unsigned*)(ws + WS_TMO);

  char* pdst = nullptr; char* pflag = nullptr;
  if (myround >= 0) {
    char* wsd = ws_all[(unsigned)pdest < (unsigned)NR ? pdest : rank];
    pdst = wsd + slot_off + (size_t)rank * (MS * KS * 2) + (size_t)(pq * 128) * 1024 + myround * 128;
    pflag = wsd + WS_FLG + (pq >> 1) * 128 + (32 * ((prem / 4 + myround) & 1) + 2 * (myround >> 1) + (pq & 1)) * 1024 + (prem / 4) * 16 + g * 4;
  }
  auto push = [&]() {
    asm volatile("global_store_dwordx4 %0, %1, %2 sc0 sc1\n\ts_nop 1" :: "v"(voffP), "v"(px0), "s"(pdst) : "memory");
    asm volatile("global_store_dwordx4 %0, %1, %2 sc0 sc1\n\ts_nop 1" :: "v"(voffP), "v"(px1), "s"(pdst + 65536) : "memory");
  };
  if (myround == 0) push();
  const bool leader = (wg == 0);
  const unsigned ldsw = lds0 + 3 * STAGE_B + g * 4;
  auto sendflag = [&]() {
    unsigned t0v, t1v;
    asm volatile("v_mov_b32 %0, 0\n\tv_mov_b32 %1, %3\n\tglobal_store_dword %0, %1, %2 sc0 sc1\n\ts_nop 1" : "=&v"(t0v), "=&v"(t1v) : "s"(pflag), "s"(call) : "memory");
  };
  if (leader) {
    unsigned t0v, t1v;
    asm volatile("v_mov_b32 %0, %2\n\tv_mov_b32 %1, %3\n\tds_write_b32 %0, %1" : "=&v"(t0v), "=&v"(t1v) : "s"(ldsw), "s"(rank * 8 + g + 4) : "memory");
  }
  WAITV(0);
  WAITL0();
  const int flagit = (myround == 0) ? -1 : myround + 1;
  BAR();
  if (myround == 0 && leader) sendflag();
  if (g == 1) BAR();

  const int fr = lane & 15, fq = lane >> 4;
  unsigned rdA = lds0 + g * STAGE_B + (unsigned)((128 * wm + fr) * 128 + ((fq ^ (fr >> 1)) * 16));
  unsigned rdB = lds0 + g * STAGE_B + (unsigned)(A_B + 2048 * fq + 1024 * wn + 128 * (fr >> 2) + 8 * (lane & 3)
                                                 + 32 * ((fr >> 3) | ((fq & 1) << 1)));

  f32x4 acc[8][4];
#pragma unroll
  for (int m = 0; m < 8; ++m)
#pragma unroll
    for (int n = 0; n < 4; ++n) acc[m][n] = f32x4{0.f, 0.f, 0.f, 0.f};

  s16x8 fa[8][2], fb[4][2];
#pragma unroll
  for (int n = 0; n < 4; ++n) fb[n][1] = s16x8{0, 0, 0, 0, 0, 0, 0, 0};
  fa[6][1] = fb[0][1]; fa[7][1] = fb[0][1];
  unsigned sread = g * STAGE_B;
  unsigned swrite = (g == 0) ? 2 * STAGE_B : 0;
  const char* abase = xloc + (g + 2) * 128;
  const char* bbase = wcol + (size_t)(rank * KS + (g + 2) * 64) * 32768;
  const char* const xl2 = xloc - (size_t)rank * (MS * KS * 2);
  const unsigned long long dstg = (unsigned long long)(stg - xl2);
  unsigned long long rem = g ? 0x552A552A552A552Aull : 0x2A552A552A552A55ull;
  unsigned long long dopoll = 0ull;
  u32x2 pollv = u32x2{0u, 0u};
  unsigned long long known = comm ? 0ull : ~0ull;

#define MF(m, n, ks) acc[m][n] = __builtin_amdgcn_mfma_f32_16x16x32_bf16(fb[n][ks], fa[m][ks], acc[m][n], 0, 0, 0)
#define RB(n, ks) { lds_char* pb0 = pbb[n] + 8192 * (ks); lds_char* pb1 = pbb[n] + (8192 * (ks) + 512); \
    s16x4 lo = __builtin_amdgcn_ds_read_tr16_b64_v4i16((__attribute__((address_space(3))) s16x4*)pb0); \
    s16x4 hi = __builtin_amdgcn_ds_read_tr16_b64_v4i16((__attribute__((address_space(3))) s16x4*)pb1); \
    fb[n][ks] = __builtin_shufflevector(lo, hi, 0, 1, 2, 3, 4, 5, 6, 7); }
#define RA0(m) fa[m][0] = *(__attribute__((address_space(3))) s16x8*)(pa0 + 2048 * (m))
#define RA1(m) fa[m][1] = *(__attribute__((address_space(3))) s16x8*)(pa1 + 2048 * (m))

  for (int i = 0; i < 32; ++i) {
    __builtin_amdgcn_s_setprio(3);
    if (dopoll != 0ull)
      asm volatile("global_load_dwordx2 %0, %1, %2 sc0 sc1" : "=v"(pollv) : "v"(voffP), "s"(flg) : "memory");
    {
      asm volatile("" : "+v"(rdA), "+v"(rdB));
      lds_char* pa0 = (lds_char*)(size_t)rdA;
      lds_char* pa1 = (lds_char*)(size_t)(rdA ^ 64u);
      lds_char* pbb[4] = {(lds_char*)(size_t)rdB, (lds_char*)(size_t)(rdB ^ 32u),
                          (lds_char*)(size_t)(rdB ^ 64u), (lds_char*)(size_t)(rdB ^ 96u)};
      const unsigned la = lds0 + swrite + wg * 8192;
      const unsigned lb = lds0 + swrite + A_B + wg * 4096;
      SCHED();
      gldsb(voffB0, bbase, lb);
      MF(6, 0, 1); MF(6, 1, 1); RB(0, 0); RB(1, 0); SCHED();
      glds128(voffB0, bbase, lb + 1024);
      MF(6, 2, 1); MF(6, 3, 1); RB(2, 0); RB(3, 0); SCHED();
      gldsb(voffB1, bbase, lb + 2048);
      MF(7, 0, 1); MF(7, 1, 1); RA0(0); RA0(1); SCHED();
      glds128(voffB1, bbase, lb + 3072);
      MF(7, 2, 1); MF(7, 3, 1); RA0(2); RA0(3); SCHED();
      glds(voffA0, abase, la);
      RA0(4); RA0(5); RA0(6); SCHED();
      glds(voffA1, abase, la + 1024);
      RA0(7); RB(0, 1); RB(1, 1); SCHED();
      glds(voffA0, abase + 16384, la + 2048);
      RB(2, 1); RB(3, 1); RA1(0); SCHED();
      glds(voffA1, abase + 16384, la + 3072);
      RA1(1); RA1(2); RA1(3); SCHED();
      glds(voffA0, abase + 32768, la + 4096);
      RA1(4); RA1(5); SCHED();
      glds(voffA1, abase + 32768, la + 5120);
      RA1(6); RA1(7); SCHED();
      glds(voffA0, abase + 49152, la + 6144);
      glds(voffA1, abase + 49152, la + 7168);
      SCHED();
    }
    int pend = 0;
    if (i == myround - 1) { push(); pend = 2; }
    if (i == flagit && leader) { sendflag(); pend = 1; }
    WAITL0();
    __builtin_amdgcn_s_setprio(0);
    BAR();
    SCHED();
    unsigned ktv;
#pragma unroll
    for (int n = 0; n < 4; ++n) MF(0, n, 0);
    SCHED();
    asm volatile("v_mov_b32 %0, %1\n\tds_read_b32 %0, %0" : "=v"(ktv) : "s"(ldsw + (unsigned)((i & 1) * 8)) : "memory");
    SCHED();
#pragma unroll
    for (int m = 1; m < 5; ++m)
#pragma unroll
      for (int n = 0; n < 4; ++n) MF(m, n, 0);
    SCHED();
    {
      asm volatile("s_waitcnt lgkmcnt(0)" : "+v"(ktv) :: "memory");
      const unsigned kt = __builtin_amdgcn_readfirstlane(ktv) & 63u;
      abase = xl2 + (((kt >> 3) != (unsigned)rank) ? dstg : 0ull) + (size_t)(((kt >> 3) << 19) | ((kt & 7u) << 7));
      bbase = wcol + (size_t)(kt << 21);
      const unsigned nx = (swrite >= STAGE_B) ? swrite - STAGE_B : swrite + 2 * STAGE_B;
      rdA += swrite - sread; rdB += swrite - sread;
      sread = swrite; swrite = nx;
    }
#pragma unroll
    for (int m = 5; m < 8; ++m)
#pragma unroll
      for (int n = 0; n < 4; ++n) MF(m, n, 0);
#pragma unroll
    for (int k = 0; k < 12; ++k) { __builtin_amdgcn_sched_group_barrier(0x008, 1, 0); __builtin_amdgcn_sched_group_barrier(0x006, 2, 0); }
    SCHED();
    asm volatile("" : "+s"(abase), "+s"(bbase), "+s"(swrite), "+v"(rdA), "+v"(rdB));
    if (leader) {
      unsigned kt = (unsigned)(rank * 8 + ((i < 1) ? g + 6 : 0));
      if (i >= 1 && i < 29) {
        unsigned long long rdy = rem;
        if (dopoll != 0ull) {
          if (pend == 0) WAITV(12); else if (pend == 1) WAITV(13); else WAITV(14);
          asm volatile("" : "+v"(pollv) :: "memory");
          unsigned long long bl = __builtin_amdgcn_ballot_w64(pollv[0] >= call && pollv[1] >= call);
          bl = bl & (bl >> 8) & 0x00FF00FF00FF00FFull; known |= bl | (bl << 8); rdy = known & rem;
          if (rdy == 0ull) {
            unsigned long long t0 = __builtin_amdgcn_s_memrealtime();
            for (;;) {
              asm volatile("global_load_dwordx2 %0, %1, %2 sc0 sc1\n\ts_waitcnt vmcnt(0)" : "=v"(pollv) : "v"(voffP), "s"(flg) : "memory");
              bl = __builtin_amdgcn_ballot_w64(pollv[0] >= call && pollv[1] >= call);
              bl = bl & (bl >> 8) & 0x00FF00FF00FF00FFull; known |= bl | (bl << 8); rdy = known & rem;
              if (rdy != 0ull) break;
              if (__builtin_amdgcn_readfirstlane(__hip_atomic_load(tmo, __ATOMIC_RELAXED, __HIP_MEMORY_SCOPE_AGENT)) == call ||
                  __builtin_amdgcn_s_memrealtime() - t0 > DEADLINE_TICKS) {
                __hip_atomic_store(tmo, call, __ATOMIC_RELAXED, __HIP_MEMORY_SCOPE_AGENT);
                rdy = rem; known = ~0ull;
                break;
              }
              __builtin_amdgcn_s_sleep(1);
            }
            WAITV(0);
          }
        }
        const unsigned t = (unsigned)__builtin_ctzll(rdy);
        rem &= ~(1ull << t);
        kt = (((unsigned)rank + t + 1u) & 7u) * 8u + (t >> 3);
      }
      unsigned t0v, t1v;
      asm volatile("v_mov_b32 %0, %2\n\tv_mov_b32 %1, %3\n\tds_write_b32 %0, %1" : "=&v"(t0v), "=&v"(t1v) : "s"(ldsw + (unsigned)((~i & 1) * 8)), "s"(kt) : "memory");
      dopoll = rem & (0ull - rem) & ~known;
      asm volatile("" : "+s"(dopoll));
    }
    SCHED();
#pragma unroll
    for (int m = 0; m < 6; ++m)
#pragma unroll
      for (int n = 0; n < 4; ++n) MF(m, n, 1);
    SCHED();
    if (pend == 0) WAITV(0); else if (pend == 1) WAITV(1); else WAITV(2);
    BAR();
    SCHED();
  }
  __builtin_amdgcn_s_setprio(3);
#pragma unroll
  for (int m = 6; m < 8; ++m)
#pragma unroll
    for (int n = 0; n < 4; ++n) MF(m, n, 1);
  __builtin_amdgcn_s_setprio(0);
  SCHED();
  if (g == 0) BAR();

  WAITV(0);
  const bool bad = comm && (__hip_atomic_load(tmo, __ATOMIC_RELAXED, __HIP_MEMORY_SCOPE_AGENT) == call);
  {
    lds_char* mine = lds + wv * 16384 + lane * 16;
#pragma unroll
    for (int m = 0; m < 4; ++m)
#pragma unroll
      for (int n = 0; n < 4; ++n) {
        f32x4 v = (g == 0) ? acc[m + 4][n] : acc[m][n];
        *(__attribute__((address_space(3))) f32x4*)(mine + (m * 4 + n) * 1024) = v;
      }
  }
  WAITL0();
  BAR();
  lds_char* theirs = lds + (wv ^ 4) * 16384;
  f32x4 fin[4][4];
#pragma unroll
  for (int m = 0; m < 4; ++m)
#pragma unroll
    for (int n = 0; n < 4; ++n) {
      f32x4 o = *(__attribute__((address_space(3))) f32x4*)(theirs + lane * 16 + (m * 4 + n) * 1024);
      f32x4 v = (g == 0) ? acc[m][n] : acc[m + 4][n];
      fin[m][n] = v + o;
    }
  WAITL0();
  SCHED();
#pragma unroll
  for (int m = 0; m < 4; ++m)
#pragma unroll
    for (int n = 0; n < 4; ++n) {
      f32x4 v = fin[m][n];
      bf16x2 lo = __builtin_convertvector(f32x2{silu_f(v[0]), silu_f(v[1])}, bf16x2);
      bf16x2 hi = __builtin_convertvector(f32x2{silu_f(v[2]), silu_f(v[3])}, bf16x2);
      u32x2 pk = u32x2{__builtin_bit_cast(unsigned, lo), __builtin_bit_cast(unsigned, hi)};
      if (bad) pk = u32x2{0x7fc07fc0u, 0x7fc07fc0u};
      const int row = 16 * m + fr, c = 2 * n + (fq >> 1);
      *(__attribute__((address_space(3))) u32x2*)(theirs + row * 128 + ((c ^ (row & 7)) * 16) + (fq & 1) * 8) = pk;
    }
  WAITL0();
  SCHED();
  {
    char* op = out + (size_t)(mh * 256 + wm * 128 + g * 64 + r8) * 32768 + (size_t)ntile * 256 + wn * 128 + (lane & 7) * 16;
#pragma unroll
    for (int i = 0; i < 8; ++i) {
      const int row = i * 8 + r8;
      u32x4 v = *(__attribute__((address_space(3))) u32x4*)(theirs + row * 128 + (((lane & 7) ^ (row & 7)) * 16));
      __builtin_nontemporal_store(v, (u32x4*)(op + (size_t)i * 8 * 32768));
    }
  }
  if (tid == 0) __hip_atomic_store((unsigned*)(ws + WS_CNT + bid * 4), call, __ATOMIC_RELAXED, __HIP_MEMORY_SCOPE_AGENT);
}

extern "C" void kernel_launch(int rank, int nranks, ncclComm_t comm,
                              hipStream_t stream,
                              void* const* d_in, const int* in_sizes, int n_in,
                              void* d_out, int out_size,
                              const int* in_dtypes, int out_dtype,
                              void* const* d_out_all,
                              void* const* d_ws_all, size_t ws_bytes) {
  if (nranks != NR || ws_bytes < WS_STG + NSLOT * SLOT_B) { fprintf(stderr, "unexpected configuration\n"); return; }
  static bool init = false;
  if (!init) {
    HIP_CHECK(hipFuncSetAttribute((const void*)a2a_gemm_silu, hipFuncAttributeMaxDynamicSharedMemorySize, LDS_B));
    init = true;
  }
  hipLaunchKernelGGL(a2a_gemm_silu, dim3(256), dim3(512), LDS_B, stream,
                     (const char*)d_in[0], (const char*)d_in[1], (char*)d_out,
                     (char* const*)d_ws_all, rank, 1);
  HIP_CHECK(hipGetLastError());
}
```

```cpp
#define BPOL ""
#include <hip/hip_runtime.h>
#include <rccl/rccl.h>
#include <cstdio>

#define HIP_CHECK(expr) do { hipError_t _e = (expr); if (_e != hipSuccess) { \
    fprintf(stderr, "HIP error %s at %s:%d\n", hipGetErrorString(_e), __FILE__, __LINE__); \
    return; } } while (0)

typedef short  s16x8 __attribute__((ext_vector_type(8)));
typedef short  s16x4 __attribute__((ext_vector_type(4)));
typedef float  f32x4 __attribute__((ext_vector_type(4)));
typedef float  f32x2 __attribute__((ext_vector_type(2)));
typedef __bf16 bf16x2 __attribute__((ext_vector_type(2)));
typedef unsigned u32x4 __attribute__((ext_vector_type(4)));
typedef unsigned u32x2 __attribute__((ext_vector_type(2)));
typedef __attribute__((address_space(3))) char lds_char;

constexpr int NR = 8;
constexpr int MS = 512, KS = 512, KT = 4096, NT = 16384;
constexpr int STAGE_B = 49152, A_B = 32768;
constexpr int LDS_B = 3 * STAGE_B + 64;
constexpr size_t WS_CNT = 0;
constexpr size_t WS_TMO = 4096;
constexpr size_t WS_FLG = 8192;
constexpr size_t WS_STG = 1u << 20;
constexpr size_t SLOT_B = (size_t)NR * MS * KS * 2;
constexpr unsigned NSLOT = 8;
constexpr unsigned long long DEADLINE_TICKS = 40000000ull;

#define WAITV(n) asm volatile("s_waitcnt vmcnt(" #n ")" ::: "memory")
#define WAITL0() asm volatile("s_waitcnt lgkmcnt(0)" ::: "memory")
#define BAR() asm volatile("s_barrier" ::: "memory")
#define SCHED() __builtin_amdgcn_sched_barrier(0)

__device__ __forceinline__ void glds(unsigned voff, const char* sbase, unsigned ldsaddr) {
  asm volatile("s_mov_b32 m0, %2\n\ts_nop 0\n\tglobal_load_lds_dwordx4 %0, %1" :: "v"(voff), "s"(sbase), "s"(ldsaddr) : "memory");
}
__device__ __forceinline__ void gldsb(unsigned voff, const char* sbase, unsigned ldsaddr) {
  asm volatile("s_mov_b32 m0, %2\n\ts_nop 0\n\tglobal_load_lds_dwordx4 %0, %1 " BPOL :: "v"(voff), "s"(sbase), "s"(ldsaddr) : "memory");
}
__device__ __forceinline__ void glds128(unsigned voff, const char* sbase, unsigned ldsaddr) {
  asm volatile("s_mov_b32 m0, %2\n\ts_nop 0\n\tglobal_load_lds_dwordx4 %0, %1 offset:128 " BPOL :: "v"(voff), "s"(sbase), "s"(ldsaddr - 128u) : "memory");
}

__device__ __forceinline__ float silu_f(float y) {
  float c = __builtin_amdgcn_fmed3f(y, -60.f, 60.f);
  float e = __builtin_amdgcn_exp2f(c * -1.4426950408889634f);
  return y * __builtin_amdgcn_rcpf(1.f + e);
}

__global__ __launch_bounds__(512, 2)
void a2a_gemm_silu(const char* __restrict__ x, const char* __restrict__ w, char* __restrict__ out,
                   char* const* __restrict__ ws_all, int rank, int mode) {
  extern __shared__ __attribute__((aligned(1024))) char smem[];
  lds_char* lds = (lds_char*)smem;
  const unsigned lds0 = (unsigned)(size_t)lds;
  const int tid = threadIdx.x, lane = tid & 63;
  const int wv = __builtin_amdgcn_readfirstlane(tid >> 6);
  const int g = wv >> 2, wg = wv & 3, wm = wg >> 1, wn = wg & 1;
  const int bid = blockIdx.x;
  const int xcd = bid & 7, jj = bid >> 3, mh = jj & 1, ntile = xcd * 16 + (jj >> 1);
  const bool comm = (mode & 1) != 0;
  char* const ws = ws_all[rank];

  const int myround = (comm && bid < 224) ? bid / 28 : -100;
  const int prem = bid % 28, pdest = (rank + 8 - (prem / 4 + 1)) & 7, pq = prem & 3;

  unsigned cntv; u32x4 px0, px1;
  const unsigned voffP = (unsigned)((tid >> 3) * 1024 + (tid & 7) * 16);
  {
    const char* cp = ws + WS_CNT + bid * 4;
    asm volatile("global_load_dword %0, %1, off sc0 sc1" : "=v"(cntv) : "v"(cp) : "memory");
    const int rr = (myround >= 0) ? myround : 0;
    const char* xp = x + (size_t)(pdest * MS + pq * 128) * 1024 + rr * 128;
    asm volatile("global_load_dwordx4 %0, %1, %2" : "=v"(px0) : "v"(voffP), "s"(xp) : "memory");
    asm volatile("global_load_dwordx4 %0, %1, %2" : "=v"(px1) : "v"(voffP), "s"(xp + 65536) : "memory");
  }

  const int r8 = lane >> 3;
  const unsigned voffA0 = (unsigned)((64 * wg + r8) * 1024 + (((lane & 7) ^ (r8 >> 1)) * 16));
  const unsigned voffA1 = (voffA0 ^ 64u) + 8192u;
  const unsigned voffB0 = (unsigned)((16 * wg + r8) * 32768 + (((lane & 7) ^ (r8 & 2)) * 16));
  const unsigned voffB1 = (voffB0 ^ 64u) + 8u * 32768u;
  const char* const xloc = x + (size_t)(rank * MS + mh * 256) * 1024;
  const char* const wcol = w + (size_t)ntile * 256;

  auto stage_tile = [&](const char* abase, const char* bbase, unsigned sbyte) {
    const unsigned la = lds0 + sbyte + wg * 8192;
#pragma unroll
    for (int j = 0; j < 4; ++j) {
      glds(voffA0, abase + j * 16384, la + j * 2048);
      glds(voffA1, abase + j * 16384, la + j * 2048 + 1024);
    }
    const unsigned lb = lds0 + sbyte + A_B + wg * 4096;
    gldsb(voffB0, bbase, lb);
    glds128(voffB0, bbase, lb + 1024);
    gldsb(voffB1, bbase, lb + 2048);
    glds128(voffB1, bbase, lb + 3072);
  };

  stage_tile(xloc + g * 128, wcol + (size_t)(rank * KS + g * 64) * 32768, g * STAGE_B);
  asm volatile("s_waitcnt vmcnt(12)" : "+v"(cntv), "+v"(px0), "+v"(px1) :: "memory");
  const unsigned call = __builtin_amdgcn_readfirstlane(cntv) + 1u;
  const size_t slot_off = WS_STG + (size_t)(call % NSLOT) * SLOT_B;
  const char* const stg = ws + slot_off + (size_t)mh * 256 * 1024;
  const char* const flg = ws + WS_FLG + mh * 128;
  unsigned* const tmo = (unsigned*)(ws + WS_TMO);

  char* pdst = nullptr; char* pflag = nullptr;
  if (myround >= 0) {
    char* wsd = ws_all[(unsigned)pdest < (unsigned)NR ? pdest : rank];
    pdst = wsd + slot_off + (size_t)rank * (MS * KS * 2) + (size_t)(pq * 128) * 1024 + myround * 128;
    pflag = wsd + WS_FLG + (pq >> 1) * 128 + (32 * ((prem / 4 + myround) & 1) + 2 * (myround >> 1) + (pq & 1)) * 1024 + (prem / 4) * 16 + g * 4;
  }
  auto push = [&]() {
    asm volatile("global_store_dwordx4 %0, %1, %2 sc0 sc1\n\ts_nop 1" :: "v"(voffP), "v"(px0), "s"(pdst) : "memory");
    asm volatile("global_store_dwordx4 %0, %1, %2 sc0 sc1\n\ts_nop 1" :: "v"(voffP), "v"(px1), "s"(pdst + 65536) : "memory");
  };
  if (myround == 0) push();
  const bool leader = (wg == 0);
  const unsigned ldsw = lds0 + 3 * STAGE_B + g * 4;
  auto sendflag = [&]() {
    unsigned t0v, t1v;
    asm volatile("v_mov_b32 %0, 0\n\tv_mov_b32 %1, %3\n\tglobal_store_dword %0, %1, %2 sc0 sc1\n\ts_nop 1" : "=&v"(t0v), "=&v"(t1v) : "s"(pflag), "s"(call) : "memory");
  };
  if (leader) {
    unsigned t0v, t1v;
    asm volatile("v_mov_b32 %0, %2\n\tv_mov_b32 %1, %3\n\tds_write_b32 %0, %1\n\tv_mov_b32 %1, 0\n\tds_write_b32 %0, %1 offset:16" : "=&v"(t0v), "=&v"(t1v) : "s"(ldsw), "s"(rank * 8 + g + 4) : "memory");
  }
  WAITV(0);
  WAITL0();
  const int flagit = (myround == 0) ? -1 : myround + 1;
  BAR();
  if (myround == 0 && leader) sendflag();
  if (g == 1) BAR();

  const int fr = lane & 15, fq = lane >> 4;
  unsigned rdA = lds0 + g * STAGE_B + (unsigned)((128 * wm + fr) * 128 + ((fq ^ (fr >> 1)) * 16));
  unsigned rdB = lds0 + g * STAGE_B + (unsigned)(A_B + 2048 * fq + 1024 * wn + 128 * (fr >> 2) + 8 * (lane & 3)
                                                 + 32 * ((fr >> 3) | ((fq & 1) << 1)));

  f32x4 acc[8][4];
#pragma unroll
  for (int m = 0; m < 8; ++m)
#pragma unroll
    for (int n = 0; n < 4; ++n) acc[m][n] = f32x4{0.f, 0.f, 0.f, 0.f};

  s16x8 fa[8][2], fb[4][2];
#pragma unroll
  for (int n = 0; n < 4; ++n) fb[n][1] = s16x8{0, 0, 0, 0, 0, 0, 0, 0};
  fa[6][1] = fb[0][1]; fa[7][1] = fb[0][1];
  unsigned sread = g * STAGE_B;
  unsigned swrite = (g == 0) ? 2 * STAGE_B : 0;
  const char* abase = xloc + (g + 2) * 128;
  const char* bbase = wcol + (size_t)(rank * KS + (g + 2) * 64) * 32768;
  const char* const xl2 = xloc - (size_t)rank * (MS * KS * 2);
  const unsigned long long dstg = (unsigned long long)(stg - xl2);
  unsigned long long rem = g ? 0x552A552A552A552Aull : 0x2A552A552A552A55ull;
  unsigned long long dopoll = 0ull;
  unsigned ktn = (unsigned)(rank * 8 + g + 6);
  u32x2 pollv = u32x2{0u, 0u};
  unsigned long long known = comm ? 0ull : ~0ull;

#define MF(m, n, ks) acc[m][n] = __builtin_amdgcn_mfma_f32_16x16x32_bf16(fb[n][ks], fa[m][ks], acc[m][n], 0, 0, 0)
#define RB(n, ks) { lds_char* pb0 = pbb[n] + 8192 * (ks); lds_char* pb1 = pbb[n] + (8192 * (ks) + 512); \
    s16x4 lo = __builtin_amdgcn_ds_read_tr16_b64_v4i16((__attribute__((address_space(3))) s16x4*)pb0); \
    s16x4 hi = __builtin_amdgcn_ds_read_tr16_b64_v4i16((__attribute__((address_space(3))) s16x4*)pb1); \
    fb[n][ks] = __builtin_shufflevector(lo, hi, 0, 1, 2, 3, 4, 5, 6, 7); }
#define RA0(m) fa[m][0] = *(__attribute__((address_space(3))) s16x8*)(pa0 + 2048 * (m))
#define RA1(m) fa[m][1] = *(__attribute__((address_space(3))) s16x8*)(pa1 + 2048 * (m))

  for (int i = 0; i < 32; ++i) {
    __builtin_amdgcn_s_setprio(3);
    if (dopoll != 0ull)
      asm volatile("global_load_dwordx2 %0, %1, %2 sc0 sc1" : "=v"(pollv) : "v"(voffP), "s"(flg) : "memory");
    {
      asm volatile("" : "+v"(rdA), "+v"(rdB));
      lds_char* pa0 = (lds_char*)(size_t)rdA;
      lds_char* pa1 = (lds_char*)(size_t)(rdA ^ 64u);
      lds_char* pbb[4] = {(lds_char*)(size_t)rdB, (lds_char*)(size_t)(rdB ^ 32u),
                          (lds_char*)(size_t)(rdB ^ 64u), (lds_char*)(size_t)(rdB ^ 96u)};
      const unsigned la = lds0 + swrite + wg * 8192;
      const unsigned lb = lds0 + swrite + A_B + wg * 4096;
      SCHED();
      gldsb(voffB0, bbase, lb);
      MF(6, 0, 1); MF(6, 1, 1); RB(0, 0); RB(1, 0); SCHED();
      glds128(voffB0, bbase, lb + 1024);
      MF(6, 2, 1); MF(6, 3, 1); RB(2, 0); RB(3, 0); SCHED();
      gldsb(voffB1, bbase, lb + 2048);
      MF(7, 0, 1); MF(7, 1, 1); RA0(0); RA0(1); SCHED();
      glds128(voffB1, bbase, lb + 3072);
      MF(7, 2, 1); MF(7, 3, 1); RA0(2); RA0(3); SCHED();
      glds(voffA0, abase, la);
      RA0(4); RA0(5); RA0(6); SCHED();
      glds(voffA1, abase, la + 1024);
      RA0(7); RB(0, 1); RB(1, 1); SCHED();
      glds(voffA0, abase + 16384, la + 2048);
      RB(2, 1); RB(3, 1); RA1(0); SCHED();
      glds(voffA1, abase + 16384, la + 3072);
      RA1(1); RA1(2); RA1(3); SCHED();
      glds(voffA0, abase + 32768, la + 4096);
      RA1(4); RA1(5); SCHED();
      glds(voffA1, abase + 32768, la + 5120);
      RA1(6); RA1(7); SCHED();
      glds(voffA0, abase + 49152, la + 6144);
      glds(voffA1, abase + 49152, la + 7168);
      SCHED();
    }
    int pend = 0;
    if (i == myround - 1) { push(); pend = 2; }
    if (i == flagit && leader) { sendflag(); pend = 1; }
    WAITL0();
    __builtin_amdgcn_s_setprio(0);
    BAR();
    SCHED();
    unsigned ktv;
#pragma unroll
    for (int n = 0; n < 4; ++n) MF(0, n, 0);
    SCHED();
    asm volatile("v_mov_b32 %0, %1\n\tds_read_b32 %0, %0" : "=v"(ktv) : "s"(ldsw + (unsigned)((i & 1) * 8)) : "memory");
    SCHED();
#pragma unroll
    for (int m = 1; m < 5; ++m)
#pragma unroll
      for (int n = 0; n < 4; ++n) MF(m, n, 0);
    SCHED();
    {
      asm volatile("s_waitcnt lgkmcnt(0)" : "+v"(ktv) :: "memory");
      const unsigned kt = __builtin_amdgcn_readfirstlane(ktv) & 63u;
      abase = xl2 + (((kt >> 3) != (unsigned)rank) ? dstg : 0ull) + (size_t)(((kt >> 3) << 19) | ((kt & 7u) << 7));
      bbase = wcol + (size_t)(kt << 21);
      const unsigned nx = (swrite >= STAGE_B) ? swrite - STAGE_B : swrite + 2 * STAGE_B;
      rdA += swrite - sread; rdB += swrite - sread;
      sread = swrite; swrite = nx;
    }
#pragma unroll
    for (int m = 5; m < 8; ++m)
#pragma unroll
      for (int n = 0; n < 4; ++n) MF(m, n, 0);
#pragma unroll
    for (int k = 0; k < 12; ++k) { __builtin_amdgcn_sched_group_barrier(0x008, 1, 0); __builtin_amdgcn_sched_group_barrier(0x006, 2, 0); }
    SCHED();
    asm volatile("" : "+s"(abase), "+s"(bbase), "+s"(swrite), "+v"(rdA), "+v"(rdB));
    if (leader) {
      if (i >= 1 && i < 29) {
        unsigned long long rdy = rem;
        if (dopoll != 0ull) {
          if (pend == 0) WAITV(12); else if (pend == 1) WAITV(13); else WAITV(14);
          asm volatile("" : "+v"(pollv) :: "memory");
          unsigned long long bl = __builtin_amdgcn_ballot_w64(pollv[0] >= call && pollv[1] >= call);
          bl = bl & (bl >> 8) & 0x00FF00FF00FF00FFull; known |= bl | (bl << 8); rdy = known & rem;
          if (rdy == 0ull) {
            unsigned long long t0 = __builtin_amdgcn_s_memrealtime();
            for (;;) {
              asm volatile("global_load_dwordx2 %0, %1, %2 sc0 sc1\n\ts_waitcnt vmcnt(0)" : "=v"(pollv) : "v"(voffP), "s"(flg) : "memory");
              bl = __builtin_amdgcn_ballot_w64(pollv[0] >= call && pollv[1] >= call);
              bl = bl & (bl >> 8) & 0x00FF00FF00FF00FFull; known |= bl | (bl << 8); rdy = known & rem;
              if (rdy != 0ull) break;
              if (__builtin_amdgcn_readfirstlane(__hip_atomic_load(tmo, __ATOMIC_RELAXED, __HIP_MEMORY_SCOPE_AGENT)) == call ||
                  __builtin_amdgcn_s_memrealtime() - t0 > DEADLINE_TICKS) {
                __hip_atomic_store(tmo, call, __ATOMIC_RELAXED, __HIP_MEMORY_SCOPE_AGENT);
                rdy = rem; known = ~0ull;
                unsigned t0v;
                asm volatile("v_mov_b32 %0, %1\n\tds_write_b32 %0, %0 offset:16" : "=&v"(t0v) : "s"(ldsw) : "memory");
                break;
              }
              __builtin_amdgcn_s_sleep(1);
            }
            WAITV(0);
          }
        }
        const unsigned t = (unsigned)__builtin_ctzll(rdy);
        rem &= ~(1ull << t);
        ktn = (((unsigned)rank + t + 1u) & 7u) * 8u + (t >> 3);
      }
      unsigned t0v, t1v;
      asm volatile("v_mov_b32 %0, %2\n\tv_mov_b32 %1, %3\n\tds_write_b32 %0, %1" : "=&v"(t0v), "=&v"(t1v) : "s"(ldsw + (unsigned)((~i & 1) * 8)), "s"(ktn) : "memory");
      dopoll = rem & (0ull - rem) & ~known;
      asm volatile("" : "+s"(dopoll));
    }
    SCHED();
#pragma unroll
    for (int m = 0; m < 6; ++m)
#pragma unroll
      for (int n = 0; n < 4; ++n) MF(m, n, 1);
    SCHED();
    if (pend == 0) WAITV(0); else if (pend == 1) WAITV(1); else WAITV(2);
    BAR();
    SCHED();
  }
  __builtin_amdgcn_s_setprio(3);
#pragma unroll
  for (int m = 6; m < 8; ++m)
#pragma unroll
    for (int n = 0; n < 4; ++n) MF(m, n, 1);
  __builtin_amdgcn_s_setprio(0);
  SCHED();
  if (g == 0) BAR();

  WAITV(0);
  const bool bad = comm && (*(volatile __attribute__((address_space(3))) unsigned long long*)(lds + 3 * STAGE_B + 16) != 0ull);
  {
    lds_char* mine = lds + wv * 16384 + lane * 16;
#pragma unroll
    for (int m = 0; m < 4; ++m)
#pragma unroll
      for (int n = 0; n < 4; ++n) {
        f32x4 v = (g == 0) ? acc[m + 4][n] : acc[m][n];
        *(__attribute__((address_space(3))) f32x4*)(mine + (m * 4 + n) * 1024) = v;
      }
  }
  WAITL0();
  BAR();
  lds_char* theirs = lds + (wv ^ 4) * 16384;
  f32x4 fin[4][4];
#pragma unroll
  for (int m = 0; m < 4; ++m)
#pragma unroll
    for (int n = 0; n < 4; ++n) {
      f32x4 o = *(__attribute__((address_space(3))) f32x4*)(theirs + lane * 16 + (m * 4 + n) * 1024);
      f32x4 v = (g == 0) ? acc[m][n] : acc[m + 4][n];
      fin[m][n] = v + o;
    }
  WAITL0();
  SCHED();
#pragma unroll
  for (int m = 0; m < 4; ++m)
#pragma unroll
    for (int n = 0; n < 4; ++n) {
      f32x4 v = fin[m][n];
      bf16x2 lo = __builtin_convertvector(f32x2{silu_f(v[0]), silu_f(v[1])}, bf16x2);
      bf16x2 hi = __builtin_convertvector(f32x2{silu_f(v[2]), silu_f(v[3])}, bf16x2);
      u32x2 pk = u32x2{__builtin_bit_cast(unsigned, lo), __builtin_bit_cast(unsigned, hi)};
      if (bad) pk = u32x2{0x7fc07fc0u, 0x7fc07fc0u};
      const int row = 16 * m + fr, c = 2 * n + (fq >> 1);
      *(__attribute__((address_space(3))) u32x2*)(theirs + row * 128 + ((c ^ (row & 7)) * 16) + (fq & 1) * 8) = pk;
    }
  WAITL0();
  SCHED();
  {
    char* op = out + (size_t)(mh * 256 + wm * 128 + g * 64 + r8) * 32768 + (size_t)ntile * 256 + wn * 128 + (lane & 7) * 16;
#pragma unroll
    for (int i = 0; i < 8; ++i) {
      const int row = i * 8 + r8;
      u32x4 v = *(__attribute__((address_space(3))) u32x4*)(theirs + row * 128 + (((lane & 7) ^ (row & 7)) * 16));
      __builtin_nontemporal_store(v, (u32x4*)(op + (size_t)i * 8 * 32768));
    }
  }
  if (tid == 0) __hip_atomic_store((unsigned*)(ws + WS_CNT + bid * 4), call, __ATOMIC_RELAXED, __HIP_MEMORY_SCOPE_AGENT);
}

extern "C" void kernel_launch(int rank, int nranks, ncclComm_t comm,
                              hipStream_t stream,
                              void* const* d_in, const int* in_sizes, int n_in,
                              void* d_out, int out_size,
                              const int* in_dtypes, int out_dtype,
                              void* const* d_out_all,
                              void* const* d_ws_all, size_t ws_bytes) {
  if (nranks != NR || ws_bytes < WS_STG + NSLOT * SLOT_B) { fprintf(stderr, "unexpected configuration\n"); return; }
  static bool init = false;
  if (!init) {
    HIP_CHECK(hipFuncSetAttribute((const void*)a2a_gemm_silu, hipFuncAttributeMaxDynamicSharedMemorySize, LDS_B));
    init = true;
  }
  hipLaunchKernelGGL(a2a_gemm_silu, dim3(256), dim3(512), LDS_B, stream,
                     (const char*)d_in[0], (const char*)d_in[1], (char*)d_out,
                     (char* const*)d_ws_all, rank, 1);
  HIP_CHECK(hipGetLastError());
}
```

```cpp
#define BPOL ""
#include <hip/hip_runtime.h>
#include <rccl/rccl.h>
#include <cstdio>

#define HIP_CHECK(expr) do { hipError_t _e = (expr); if (_e != hipSuccess) { \
    fprintf(stderr, "HIP error %s at %s:%d\n", hipGetErrorString(_e), __FILE__, __LINE__); \
    return; } } while (0)

typedef short  s16x8 __attribute__((ext_vector_type(8)));
typedef short  s16x4 __attribute__((ext_vector_type(4)));
typedef float  f32x4 __attribute__((ext_vector_type(4)));
typedef float  f32x2 __attribute__((ext_vector_type(2)));
typedef __bf16 bf16x2 __attribute__((ext_vector_type(2)));
typedef unsigned u32x4 __attribute__((ext_vector_type(4)));
typedef unsigned u32x2 __attribute__((ext_vector_type(2)));
typedef __attribute__((address_space(3))) char lds_char;

constexpr int NR = 8;
constexpr int MS = 512, KS = 512, KT = 4096, NT = 16384;
constexpr int STAGE_B = 49152, A_B = 32768;
constexpr int LDS_B = 3 * STAGE_B + 64;
constexpr size_t WS_CNT = 0;
constexpr size_t WS_TMO = 4096;
constexpr size_t WS_FLG = 8192;
constexpr size_t WS_STG = 1u << 20;
constexpr size_t SLOT_B = (size_t)NR * MS * KS * 2;
constexpr unsigned NSLOT = 8;
constexpr unsigned long long DEADLINE_TICKS = 40000000ull;

#define WAITV(n) asm volatile("s_waitcnt vmcnt(" #n ")" ::: "memory")
#define WAITL0() asm volatile("s_waitcnt lgkmcnt(0)" ::: "memory")
#define BAR() asm volatile("s_barrier" ::: "memory")
#define SCHED() __builtin_amdgcn_sched_barrier(0)

__device__ __forceinline__ void glds(unsigned voff, const char* sbase, unsigned ldsaddr) {
  asm volatile("s_mov_b32 m0, %2\n\ts_nop 0\n\tglobal_load_lds_dwordx4 %0, %1" :: "v"(voff), "s"(sbase), "s"(ldsaddr) : "memory");
}
__device__ __forceinline__ void gldsb(unsigned voff, const char* sbase, unsigned ldsaddr) {
  asm volatile("s_mov_b32 m0, %2\n\ts_nop 0\n\tglobal_load_lds_dwordx4 %0, %1 " BPOL :: "v"(voff), "s"(sbase), "s"(ldsaddr) : "memory");
}
__device__ __forceinline__ void glds128(unsigned voff, const char* sbase, unsigned ldsaddr) {
  asm volatile("s_mov_b32 m0, %2\n\ts_nop 0\n\tglobal_load_lds_dwordx4 %0, %1 offset:128 " BPOL :: "v"(voff), "s"(sbase), "s"(ldsaddr - 128u) : "memory");
}

__device__ __forceinline__ float silu_f(float y) {
  float c = __builtin_amdgcn_fmed3f(y, -60.f, 60.f);
  float e = __builtin_amdgcn_exp2f(c * -1.4426950408889634f);
  return y * __builtin_amdgcn_rcpf(1.f + e);
}

__global__ __launch_bounds__(512, 2)
void a2a_gemm_silu(const char* __restrict__ x, const char* __restrict__ w, char* __restrict__ out,
                   char* const* __restrict__ ws_all, int rank, int mode) {
  extern __shared__ __attribute__((aligned(1024))) char smem[];
  lds_char* lds = (lds_char*)smem;
  const unsigned lds0 = (unsigned)(size_t)lds;
  const int tid = threadIdx.x, lane = tid & 63;
  const int wv = __builtin_amdgcn_readfirstlane(tid >> 6);
  const int g = wv >> 2, wg = wv & 3, wm = wg >> 1, wn = wg & 1;
  const int bid = blockIdx.x;
  const int xcd = bid & 7, jj = bid >> 3, mh = jj & 1, ntile = xcd * 16 + (jj >> 1);
  const bool comm = (mode & 1) != 0;
  char* const ws = ws_all[rank];

  const int myround = (comm && bid < 224) ? bid / 28 : -100;
  const int prem = bid % 28, pdest = (rank + 8 - (prem / 4 + 1)) & 7, pq = prem & 3;

  unsigned cntv; u32x4 px0, px1;
  const unsigned voffP = (unsigned)((tid >> 3) * 1024 + (tid & 7) * 16);
  {
    const char* cp = ws + WS_CNT + bid * 4;
    asm volatile("global_load_dword %0, %1, off sc0 sc1" : "=v"(cntv) : "v"(cp) : "memory");
    const int rr = (myround >= 0) ? myround : 0;
    const char* xp = x + (size_t)(pdest * MS + pq * 128) * 1024 + rr * 128;
    asm volatile("global_load_dwordx4 %0, %1, %2" : "=v"(px0) : "v"(voffP), "s"(xp) : "memory");
    asm volatile("global_load_dwordx4 %0, %1, %2" : "=v"(px1) : "v"(voffP), "s"(xp + 65536) : "memory");
  }

  const int r8 = lane >> 3;
  const unsigned voffA0 = (unsigned)((64 * wg + r8) * 1024 + (((lane & 7) ^ (r8 >> 1)) * 16));
  const unsigned voffA1 = (voffA0 ^ 64u) + 8192u;
  const unsigned voffB0 = (unsigned)((16 * wg + r8) * 32768 + (((lane & 7) ^ (r8 & 2)) * 16));
  const unsigned voffB1 = (voffB0 ^ 64u) + 8u * 32768u;
  const char* const xloc = x + (size_t)(rank * MS + mh * 256) * 1024;
  const char* const wcol = w + (size_t)ntile * 256;

  auto stage_tile = [&](const char* abase, const char* bbase, unsigned sbyte) {
    const unsigned la = lds0 + sbyte + wg * 8192;
#pragma unroll
    for (int j = 0; j < 4; ++j) {
      glds(voffA0, abase + j * 16384, la + j * 2048);
      glds(voffA1, abase + j * 16384, la + j * 2048 + 1024);
    }
    const unsigned lb = lds0 + sbyte + A_B + wg * 4096;
    gldsb(voffB0, bbase, lb);
    glds128(voffB0, bbase, lb + 1024);
    gldsb(voffB1, bbase, lb + 2048);
    glds128(voffB1, bbase, lb + 3072);
  };

  stage_tile(xloc + g * 128, wcol + (size_t)(rank * KS + g * 64) * 32768, g * STAGE_B);
  asm volatile("s_waitcnt vmcnt(12)" : "+v"(cntv), "+v"(px0), "+v"(px1) :: "memory");
  const unsigned call = __builtin_amdgcn_readfirstlane(cntv) + 1u;
  const size_t slot_off = WS_STG + (size_t)(call % NSLOT) * SLOT_B;
  const char* const stg = ws + slot_off + (size_t)mh * 256 * 1024;
  const char* const flg = ws + WS_FLG + (mh * 2 + g) * 512;
  const unsigned voffF = (unsigned)lane * 8u;
  unsigned* const tmo = (unsigned*)(ws + WS_TMO);

  char* pdst = nullptr; char* pflag = nullptr;
  if (myround >= 0) {
    char* wsd = ws_all[(unsigned)pdest < (unsigned)NR ? pdest : rank];
    pdst = wsd + slot_off + (size_t)rank * (MS * KS * 2) + (size_t)(pq * 128) * 1024 + myround * 128;
    pflag = wsd + WS_FLG + ((pq >> 1) * 2 + ((prem / 4 + myround) & 1)) * 512 + ((myround >> 1) * 16 + (pq & 1) * 8 + prem / 4) * 8 + g * 4;
  }
  auto push = [&]() {
    asm volatile("global_store_dwordx4 %0, %1, %2 sc0 sc1\n\ts_nop 1" :: "v"(voffP), "v"(px0), "s"(pdst) : "memory");
    asm volatile("global_store_dwordx4 %0, %1, %2 sc0 sc1\n\ts_nop 1" :: "v"(voffP), "v"(px1), "s"(pdst + 65536) : "memory");
  };
  if (myround == 0) push();
  const bool leader = (wg == 0);
  const unsigned ldsw = lds0 + 3 * STAGE_B + g * 4;
  auto sendflag = [&]() {
    unsigned t0v, t1v;
    asm volatile("v_mov_b32 %0, 0\n\tv_mov_b32 %1, %3\n\tglobal_store_dword %0, %1, %2 sc0 sc1\n\ts_nop 1" : "=&v"(t0v), "=&v"(t1v) : "s"(pflag), "s"(call) : "memory");
  };
  if (leader) {
    unsigned t0v, t1v;
    asm volatile("v_mov_b32 %0, %2\n\tv_mov_b32 %1, %3\n\tds_write_b32 %0, %1\n\tv_mov_b32 %1, 0\n\tds_write_b32 %0, %1 offset:16" : "=&v"(t0v), "=&v"(t1v) : "s"(ldsw), "s"(rank * 8 + g + 4) : "memory");
  }
  WAITV(0);
  WAITL0();
  const int flagit = (myround == 0) ? -1 : myround + 1;
  BAR();
  if (myround == 0 && leader) sendflag();
  if (g == 1) BAR();

  const int fr = lane & 15, fq = lane >> 4;
  unsigned rdA = lds0 + g * STAGE_B + (unsigned)((128 * wm + fr) * 128 + ((fq ^ (fr >> 1)) * 16));
  unsigned rdB = lds0 + g * STAGE_B + (unsigned)(A_B + 2048 * fq + 1024 * wn + 128 * (fr >> 2) + 8 * (lane & 3)
                                                 + 32 * ((fr >> 3) | ((fq & 1) << 1)));

  f32x4 acc[8][4];
#pragma unroll
  for (int m = 0; m < 8; ++m)
#pragma unroll
    for (int n = 0; n < 4; ++n) acc[m][n] = f32x4{0.f, 0.f, 0.f, 0.f};

  s16x8 fa[8][2], fb[4][2];
#pragma unroll
  for (int n = 0; n < 4; ++n) fb[n][1] = s16x8{0, 0, 0, 0, 0, 0, 0, 0};
  fa[6][1] = fb[0][1]; fa[7][1] = fb[0][1];
  unsigned sread = g * STAGE_B;
  unsigned swrite = (g == 0) ? 2 * STAGE_B : 0;
  const char* abase = xloc + (g + 2) * 128;
  const char* bbase = wcol + (size_t)(rank * KS + (g + 2) * 64) * 32768;
  const char* const xl2 = xloc - (size_t)rank * (MS * KS * 2);
  const unsigned long long dstg = (unsigned long long)(stg - xl2);
  unsigned long long rem = g ? 0x552A552A552A552Aull : 0x2A552A552A552A55ull;
  unsigned long long dopoll = 0ull;
  unsigned ktn = (unsigned)(rank * 8 + g + 6);
  u32x2 pollv = u32x2{0u, 0u};
  unsigned long long known = comm ? 0ull : ~0ull;

#define MF(m, n, ks) acc[m][n] = __builtin_amdgcn_mfma_f32_16x16x32_bf16(fb[n][ks], fa[m][ks], acc[m][n], 0, 0, 0)
#define RB(n, ks) { lds_char* pb0 = pbb[n] + 8192 * (ks); lds_char* pb1 = pbb[n] + (8192 * (ks) + 512); \
    s16x4 lo = __builtin_amdgcn_ds_read_tr16_b64_v4i16((__attribute__((address_space(3))) s16x4*)pb0); \
    s16x4 hi = __builtin_amdgcn_ds_read_tr16_b64_v4i16((__attribute__((address_space(3))) s16x4*)pb1); \
    fb[n][ks] = __builtin_shufflevector(lo, hi, 0, 1, 2, 3, 4, 5, 6, 7); }
#define RA0(m) fa[m][0] = *(__attribute__((address_space(3))) s16x8*)(pa0 + 2048 * (m))
#define RA1(m) fa[m][1] = *(__attribute__((address_space(3))) s16x8*)(pa1 + 2048 * (m))

  for (int i = 0; i < 32; ++i) {
    __builtin_amdgcn_s_setprio(3);
    if (dopoll != 0ull)
      asm volatile("global_load_dwordx2 %0, %1, %2 sc0 sc1" : "=v"(pollv) : "v"(voffF), "s"(flg) : "memory");
    {
      asm volatile("" : "+v"(rdA), "+v"(rdB));
      lds_char* pa0 = (lds_char*)(size_t)rdA;
      lds_char* pa1 = (lds_char*)(size_t)(rdA ^ 64u);
      lds_char* pbb[4] = {(lds_char*)(size_t)rdB, (lds_char*)(size_t)(rdB ^ 32u),
                          (lds_char*)(size_t)(rdB ^ 64u), (lds_char*)(size_t)(rdB ^ 96u)};
      const unsigned la = lds0 + swrite + wg * 8192;
      const unsigned lb = lds0 + swrite + A_B + wg * 4096;
      SCHED();
      gldsb(voffB0, bbase, lb);
      MF(6, 0, 1); MF(6, 1, 1); RB(0, 0); RB(1, 0); SCHED();
      glds128(voffB0, bbase, lb + 1024);
      MF(6, 2, 1); MF(6, 3, 1); RB(2, 0); RB(3, 0); SCHED();
      gldsb(voffB1, bbase, lb + 2048);
      MF(7, 0, 1); MF(7, 1, 1); RA0(0); RA0(1); SCHED();
      glds128(voffB1, bbase, lb + 3072);
      MF(7, 2, 1); MF(7, 3, 1); RA0(2); RA0(3); SCHED();
      glds(voffA0, abase, la);
      RA0(4); RA0(5); RA0(6); SCHED();
      glds(voffA1, abase, la + 1024);
      RA0(7); RB(0, 1); RB(1, 1); SCHED();
      glds(voffA0, abase + 16384, la + 2048);
      RB(2, 1); RB(3, 1); RA1(0); SCHED();
      glds(voffA1, abase + 16384, la + 3072);
      RA1(1); RA1(2); RA1(3); SCHED();
      glds(voffA0, abase + 32768, la + 4096);
      RA1(4); RA1(5); SCHED();
      glds(voffA1, abase + 32768, la + 5120);
      RA1(6); RA1(7); SCHED();
      glds(voffA0, abase + 49152, la + 6144);
      glds(voffA1, abase + 49152, la + 7168);
      SCHED();
    }
    int pend = 0;
    if (i == myround - 1) { push(); pend = 2; }
    if (i == flagit && leader) { sendflag(); pend = 1; }
    WAITL0();
    __builtin_amdgcn_s_setprio(0);
    BAR();
    SCHED();
    unsigned ktv;
#pragma unroll
    for (int n = 0; n < 4; ++n) MF(0, n, 0);
    SCHED();
    asm volatile("v_mov_b32 %0, %1\n\tds_read_b32 %0, %0" : "=v"(ktv) : "s"(ldsw + (unsigned)((i & 1) * 8)) : "memory");
    SCHED();
#pragma unroll
    for (int m = 1; m < 5; ++m)
#pragma unroll
      for (int n = 0; n < 4; ++n) MF(m, n, 0);
    SCHED();
    {
      asm volatile("s_waitcnt lgkmcnt(0)" : "+v"(ktv) :: "memory");
      const unsigned kt = __builtin_amdgcn_readfirstlane(ktv) & 63u;
      abase = xl2 + (((kt >> 3) != (unsigned)rank) ? dstg : 0ull) + (size_t)(((kt >> 3) << 19) | ((kt & 7u) << 7));
      bbase = wcol + (size_t)(kt << 21);
      const unsigned nx = (swrite >= STAGE_B) ? swrite - STAGE_B : swrite + 2 * STAGE_B;
      rdA += swrite - sread; rdB += swrite - sread;
      sread = swrite; swrite = nx;
    }
#pragma unroll
    for (int m = 5; m < 8; ++m)
#pragma unroll
      for (int n = 0; n < 4; ++n) MF(m, n, 0);
#pragma unroll
    for (int k = 0; k < 12; ++k) { __builtin_amdgcn_sched_group_barrier(0x008, 1, 0); __builtin_amdgcn_sched_group_barrier(0x006, 2, 0); }
    SCHED();
    asm volatile("" : "+s"(abase), "+s"(bbase), "+s"(swrite), "+v"(rdA), "+v"(rdB));
    if (leader) {
      if (i >= 1 && i < 29) {
        unsigned long long rdy = rem & known;
        if (dopoll != 0ull) {
          if (pend == 0) WAITV(12); else if (pend == 1) WAITV(13); else WAITV(14);
          asm volatile("" : "+v"(pollv) :: "memory");
          unsigned long long bl = __builtin_amdgcn_ballot_w64(pollv[0] >= call && pollv[1] >= call);
          bl = bl & (bl >> 8) & 0x00FF00FF00FF00FFull; known |= bl | (bl << 8); rdy = known & rem;
          if (rdy == 0ull) {
            unsigned long long t0 = __builtin_amdgcn_s_memrealtime();
            for (;;) {
              asm volatile("global_load_dwordx2 %0, %1, %2 sc0 sc1\n\ts_waitcnt vmcnt(0)" : "=v"(pollv) : "v"(voffF), "s"(flg) : "memory");
              bl = __builtin_amdgcn_ballot_w64(pollv[0] >= call && pollv[1] >= call);
              bl = bl & (bl >> 8) & 0x00FF00FF00FF00FFull; known |= bl | (bl << 8); rdy = known & rem;
              if (rdy != 0ull) break;
              if (__builtin_amdgcn_readfirstlane(__hip_atomic_load(tmo, __ATOMIC_RELAXED, __HIP_MEMORY_SCOPE_AGENT)) == call ||
                  __builtin_amdgcn_s_memrealtime() - t0 > DEADLINE_TICKS) {
                __hip_atomic_store(tmo, call, __ATOMIC_RELAXED, __HIP_MEMORY_SCOPE_AGENT);
                rdy = rem; known = ~0ull;
                unsigned t0v;
                asm volatile("v_mov_b32 %0, %1\n\tds_write_b32 %0, %0 offset:16" : "=&v"(t0v) : "s"(ldsw) : "memory");
                break;
              }
              __builtin_amdgcn_s_sleep(1);
            }
            WAITV(0);
          }
        }
        const unsigned t = (unsigned)__builtin_ctzll(rdy);
        rem &= ~(1ull << t);
        ktn = (((unsigned)rank + t + 1u) & 7u) * 8u + (t >> 3);
      }
      unsigned t0v, t1v;
      asm volatile("v_mov_b32 %0, %2\n\tv_mov_b32 %1, %3\n\tds_write_b32 %0, %1" : "=&v"(t0v), "=&v"(t1v) : "s"(ldsw + (unsigned)((~i & 1) * 8)), "s"(ktn) : "memory");
      {
        const unsigned long long kr = rem & known, unk = rem & ~known;
        const unsigned long long pa = (kr == 0ull) ? unk : 0ull;
        const unsigned long long pb = ((i & 3) == 3) ? (unk & (0ull - rem) & ~(rem - 1ull)) : 0ull;
        dopoll = pa | pb;
      }
      asm volatile("" : "+s"(dopoll));
    }
    SCHED();
#pragma unroll
    for (int m = 0; m < 6; ++m)
#pragma unroll
      for (int n = 0; n < 4; ++n) MF(m, n, 1);
    SCHED();
    if (pend == 0) WAITV(0); else if (pend == 1) WAITV(1); else WAITV(2);
    BAR();
    SCHED();
  }
  __builtin_amdgcn_s_setprio(3);
#pragma unroll
  for (int m = 6; m < 8; ++m)
#pragma unroll
    for (int n = 0; n < 4; ++n) MF(m, n, 1);
  __builtin_amdgcn_s_setprio(0);
  SCHED();
  if (g == 0) BAR();

  WAITV(0);
  const bool bad = comm && (*(volatile __attribute__((address_space(3))) unsigned long long*)(lds + 3 * STAGE_B + 16) != 0ull);
  {
    lds_char* mine = lds + wv * 16384 + lane * 16;
#pragma unroll
    for (int m = 0; m < 4; ++m)
#pragma unroll
      for (int n = 0; n < 4; ++n) {
        f32x4 v = (g == 0) ? acc[m + 4][n] : acc[m][n];
        *(__attribute__((address_space(3))) f32x4*)(mine + (m * 4 + n) * 1024) = v;
      }
  }
  WAITL0();
  BAR();
  lds_char* theirs = lds + (wv ^ 4) * 16384;
  f32x4 fin[4][4];
#pragma unroll
  for (int m = 0; m < 4; ++m)
#pragma unroll
    for (int n = 0; n < 4; ++n) {
      f32x4 o = *(__attribute__((address_space(3))) f32x4*)(theirs + lane * 16 + (m * 4 + n) * 1024);
      f32x4 v = (g == 0) ? acc[m][n] : acc[m + 4][n];
      fin[m][n] = v + o;
    }
  WAITL0();
  SCHED();
#pragma unroll
  for (int m = 0; m < 4; ++m)
#pragma unroll
    for (int n = 0; n < 4; ++n) {
      f32x4 v = fin[m][n];
      bf16x2 lo = __builtin_convertvector(f32x2{silu_f(v[0]), silu_f(v[1])}, bf16x2);
      bf16x2 hi = __builtin_convertvector(f32x2{silu_f(v[2]), silu_f(v[3])}, bf16x2);
      u32x2 pk = u32x2{__builtin_bit_cast(unsigned, lo), __builtin_bit_cast(unsigned, hi)};
      if (bad) pk = u32x2{0x7fc07fc0u, 0x7fc07fc0u};
      const int row = 16 * m + fr, c = 2 * n + (fq >> 1);
      *(__attribute__((address_space(3))) u32x2*)(theirs + row * 128 + ((c ^ (row & 7)) * 16) + (fq & 1) * 8) = pk;
    }
  WAITL0();
  SCHED();
  {
    char* op = out + (size_t)(mh * 256 + wm * 128 + g * 64 + r8) * 32768 + (size_t)ntile * 256 + wn * 128 + (lane & 7) * 16;
#pragma unroll
    for (int i = 0; i < 8; ++i) {
      const int row = i * 8 + r8;
      u32x4 v = *(__attribute__((address_space(3))) u32x4*)(theirs + row * 128 + (((lane & 7) ^ (row & 7)) * 16));
      __builtin_nontemporal_store(v, (u32x4*)(op + (size_t)i * 8 * 32768));
    }
  }
  if (tid == 0) __hip_atomic_store((unsigned*)(ws + WS_CNT + bid * 4), call, __ATOMIC_RELAXED, __HIP_MEMORY_SCOPE_AGENT);
}

extern "C" void kernel_launch(int rank, int nranks, ncclComm_t comm,
                              hipStream_t stream,
                              void* const* d_in, const int* in_sizes, int n_in,
                              void* d_out, int out_size,
                              const int* in_dtypes, int out_dtype,
                              void* const* d_out_all,
                              void* const* d_ws_all, size_t ws_bytes) {
  if (nranks != NR || ws_bytes < WS_STG + NSLOT * SLOT_B) { fprintf(stderr, "unexpected configuration\n"); return; }
  static bool init = false;
  if (!init) {
    HIP_CHECK(hipFuncSetAttribute((const void*)a2a_gemm_silu, hipFuncAttributeMaxDynamicSharedMemorySize, LDS_B));
    init = true;
  }
  hipLaunchKernelGGL(a2a_gemm_silu, dim3(256), dim3(512), LDS_B, stream,
                     (const char*)d_in[0], (const char*)d_in[1], (char*)d_out,
                     (char* const*)d_ws_all, rank, 1);
  HIP_CHECK(hipGetLastError());
}
```

```cpp
#define BPOL ""
#include <hip/hip_runtime.h>
#include <rccl/rccl.h>
#include <cstdio>

#define HIP_CHECK(expr) do { hipError_t _e = (expr); if (_e != hipSuccess) { \
    fprintf(stderr, "HIP error %s at %s:%d\n", hipGetErrorString(_e), __FILE__, __LINE__); \
    return; } } while (0)

typedef short  s16x8 __attribute__((ext_vector_type(8)));
typedef short  s16x4 __attribute__((ext_vector_type(4)));
typedef float  f32x4 __attribute__((ext_vector_type(4)));
typedef float  f32x2 __attribute__((ext_vector_type(2)));
typedef __bf16 bf16x2 __attribute__((ext_vector_type(2)));
typedef unsigned u32x4 __attribute__((ext_vector_type(4)));
typedef unsigned u32x2 __attribute__((ext_vector_type(2)));
typedef __attribute__((address_space(3))) char lds_char;

constexpr int NR = 8;
constexpr int MS = 512, KS = 512, KT = 4096, NT = 16384;
constexpr int STAGE_B = 49152, A_B = 32768;
constexpr int LDS_B = 3 * STAGE_B + 64;
constexpr size_t WS_CNT = 0;
constexpr size_t WS_TMO = 4096;
constexpr size_t WS_FLG = 8192;
constexpr size_t WS_STG = 1u << 20;
constexpr size_t SLOT_B = (size_t)NR * MS * KS * 2;
constexpr unsigned NSLOT = 8;
constexpr unsigned long long DEADLINE_TICKS = 40000000ull;

#define WAITV(n) asm volatile("s_waitcnt vmcnt(" #n ")" ::: "memory")
#define WAITL0() asm volatile("s_waitcnt lgkmcnt(0)" ::: "memory")
#define BAR() asm volatile("s_barrier" ::: "memory")
#define SCHED() __builtin_amdgcn_sched_barrier(0)

__device__ __forceinline__ void glds(unsigned voff, const char* sbase, unsigned ldsaddr) {
  asm volatile("s_mov_b32 m0, %2\n\ts_nop 0\n\tglobal_load_lds_dwordx4 %0, %1" :: "v"(voff), "s"(sbase), "s"(ldsaddr) : "memory");
}
__device__ __forceinline__ void gldsb(unsigned voff, const char* sbase, unsigned ldsaddr) {
  asm volatile("s_mov_b32 m0, %2\n\ts_nop 0\n\tglobal_load_lds_dwordx4 %0, %1 " BPOL :: "v"(voff), "s"(sbase), "s"(ldsaddr) : "memory");
}
__device__ __forceinline__ void glds128(unsigned voff, const char* sbase, unsigned ldsaddr) {
  asm volatile("s_mov_b32 m0, %2\n\ts_nop 0\n\tglobal_load_lds_dwordx4 %0, %1 offset:128 " BPOL :: "v"(voff), "s"(sbase), "s"(ldsaddr - 128u) : "memory");
}

__device__ __forceinline__ float silu_f(float y) {
  float c = __builtin_amdgcn_fmed3f(y, -60.f, 60.f);
  float e = __builtin_amdgcn_exp2f(c * -1.4426950408889634f);
  return y * __builtin_amdgcn_rcpf(1.f + e);
}

__global__ __launch_bounds__(512, 2)
void a2a_gemm_silu(const char* __restrict__ x, const char* __restrict__ w, char* __restrict__ out,
                   char* const* __restrict__ ws_all, int rank, int mode) {
  extern __shared__ __attribute__((aligned(1024))) char smem[];
  lds_char* lds = (lds_char*)smem;
  const unsigned lds0 = (unsigned)(size_t)lds;
  const int tid = threadIdx.x, lane = tid & 63;
  const int wv = __builtin_amdgcn_readfirstlane(tid >> 6);
  const int g = wv >> 2, wg = wv & 3, wm = wg >> 1, wn = wg & 1;
  const int bid = blockIdx.x;
  const int xcd = bid & 7, jj = bid >> 3, mh = jj & 1, ntile = xcd * 16 + (jj >> 1);
  const bool comm = (mode & 1) != 0;
  char* const ws = ws_all[rank];

  const int myround = (comm && bid < 224) ? bid / 28 : -100;
  const int prem = bid % 28, pdest = (rank + 8 - (prem / 4 + 1)) & 7, pq = prem & 3;

  unsigned cntv; u32x4 px0, px1;
  const unsigned voffP = (unsigned)((tid >> 3) * 1024 + (tid & 7) * 16);
  {
    const char* cp = ws + WS_CNT + bid * 4;
    asm volatile("global_load_dword %0, %1, off sc0 sc1" : "=v"(cntv) : "v"(cp) : "memory");
    const int rr = (myround >= 0) ? myround : 0;
    const char* xp = x + (size_t)(pdest * MS + pq * 128) * 1024 + rr * 128;
    asm volatile("global_load_dwordx4 %0, %1, %2" : "=v"(px0) : "v"(voffP), "s"(xp) : "memory");
    asm volatile("global_load_dwordx4 %0, %1, %2" : "=v"(px1) : "v"(voffP), "s"(xp + 65536) : "memory");
  }

  const int r8 = lane >> 3;
  const unsigned voffA0 = (unsigned)((64 * wg + r8) * 1024 + (((lane & 7) ^ (r8 >> 1)) * 16));
  const unsigned voffA1 = (voffA0 ^ 64u) + 8192u;
  const unsigned voffB0 = (unsigned)((16 * wg + r8) * 32768 + (((lane & 7) ^ (r8 & 2)) * 16));
  const unsigned voffB1 = (voffB0 ^ 64u) + 8u * 32768u;
  const char* const xloc = x + (size_t)(rank * MS + mh * 256) * 1024;
  const char* const wcol = w + (size_t)ntile * 256;

  auto stage_tile = [&](const char* abase, const char* bbase, unsigned sbyte) {
    const unsigned la = lds0 + sbyte + wg * 8192;
#pragma unroll
    for (int j = 0; j < 4; ++j) {
      glds(voffA0, abase + j * 16384, la + j * 2048);
      glds(voffA1, abase + j * 16384, la + j * 2048 + 1024);
    }
    const unsigned lb = lds0 + sbyte + A_B + wg * 4096;
    gldsb(voffB0, bbase, lb);
    glds128(voffB0, bbase, lb + 1024);
    gldsb(voffB1, bbase, lb + 2048);
    glds128(voffB1, bbase, lb + 3072);
  };

  stage_tile(xloc + g * 128, wcol + (size_t)(rank * KS + g * 64) * 32768, g * STAGE_B);
  asm volatile("s_waitcnt vmcnt(12)" : "+v"(cntv), "+v"(px0), "+v"(px1) :: "memory");
  const unsigned call = __builtin_amdgcn_readfirstlane(cntv) + 1u;
  const size_t slot_off = WS_STG + (size_t)(call % NSLOT) * SLOT_B;
  const char* const stg = ws + slot_off + (size_t)mh * 256 * 1024;
  const char* const flg = ws + WS_FLG + (mh * 2 + g) * 512;
  const unsigned voffF = (unsigned)lane * 8u;
  unsigned* const tmo = (unsigned*)(ws + WS_TMO);

  char* pdst = nullptr; char* pflag = nullptr;
  if (myround >= 0) {
    char* wsd = ws_all[(unsigned)pdest < (unsigned)NR ? pdest : rank];
    pdst = wsd + slot_off + (size_t)rank * (MS * KS * 2) + (size_t)(pq * 128) * 1024 + myround * 128;
    pflag = wsd + WS_FLG + ((pq >> 1) * 2 + ((prem / 4 + myround) & 1)) * 512 + ((myround >> 1) * 16 + (pq & 1) * 8 + prem / 4) * 8 + g * 4;
  }
  auto push = [&]() {
    asm volatile("global_store_dwordx4 %0, %1, %2 sc0 sc1\n\ts_nop 1" :: "v"(voffP), "v"(px0), "s"(pdst) : "memory");
    asm volatile("global_store_dwordx4 %0, %1, %2 sc0 sc1\n\ts_nop 1" :: "v"(voffP), "v"(px1), "s"(pdst + 65536) : "memory");
  };
  if (myround == 0) push();
  const bool leader = (wg == 0);
  const unsigned ldsw = lds0 + 3 * STAGE_B + g * 4;
  auto sendflag = [&]() {
    unsigned t0v, t1v;
    asm volatile("v_mov_b32 %0, 0\n\tv_mov_b32 %1, %3\n\tglobal_store_dword %0, %1, %2 sc0 sc1\n\ts_nop 1" : "=&v"(t0v), "=&v"(t1v) : "s"(pflag), "s"(call) : "memory");
  };
  if (leader) {
    unsigned t0v, t1v;
    asm volatile("v_mov_b32 %0, %2\n\tv_mov_b32 %1, %3\n\tds_write_b32 %0, %1\n\tv_mov_b32 %1, 0\n\tds_write_b32 %0, %1 offset:16" : "=&v"(t0v), "=&v"(t1v) : "s"(ldsw), "s"(rank * 8 + g + 4) : "memory");
  }
  WAITV(0);
  WAITL0();
  const int flagit = (myround == 0) ? -1 : myround + 1;
  BAR();
  if (myround == 0 && leader) sendflag();
  if (g == 1) BAR();

  const int fr = lane & 15, fq = lane >> 4;
  unsigned rdA = lds0 + g * STAGE_B + (unsigned)((128 * wm + fr) * 128 + ((fq ^ (fr >> 1)) * 16));
  unsigned rdB = lds0 + g * STAGE_B + (unsigned)(A_B + 2048 * fq + 1024 * wn + 128 * (fr >> 2) + 8 * (lane & 3)
                                                 + 32 * ((fr >> 3) | ((fq & 1) << 1)));

  f32x4 acc[8][4];
#pragma unroll
  for (int m = 0; m < 8; ++m)
#pragma unroll
    for (int n = 0; n < 4; ++n) acc[m][n] = f32x4{0.f, 0.f, 0.f, 0.f};

  s16x8 fa[8][2], fb[4][2];
#pragma unroll
  for (int n = 0; n < 4; ++n) fb[n][1] = s16x8{0, 0, 0, 0, 0, 0, 0, 0};
  fa[6][1] = fb[0][1]; fa[7][1] = fb[0][1];
  unsigned sread = g * STAGE_B;
  unsigned swrite = (g == 0) ? 2 * STAGE_B : 0;
  const char* abase = xloc + (g + 2) * 128;
  const char* bbase = wcol + (size_t)(rank * KS + (g + 2) * 64) * 32768;
  const char* const xl2 = xloc - (size_t)rank * (MS * KS * 2);
  const unsigned long long dstg = (unsigned long long)(stg - xl2);
  unsigned long long rem = g ? 0x552A552A552A552Aull : 0x2A552A552A552A55ull;
  unsigned long long dopoll = 0ull;
  unsigned ktn = (unsigned)(rank * 8 + g + 6);
  u32x2 pollv = u32x2{0u, 0u};
  unsigned long long known = comm ? 0ull : ~0ull;

#define MF(m, n, ks) acc[m][n] = __builtin_amdgcn_mfma_f32_16x16x32_bf16(fb[n][ks], fa[m][ks], acc[m][n], 0, 0, 0)
#define RB(n, ks) { lds_char* pb0 = pbb[n] + 8192 * (ks); lds_char* pb1 = pbb[n] + (8192 * (ks) + 512); \
    s16x4 lo = __builtin_amdgcn_ds_read_tr16_b64_v4i16((__attribute__((address_space(3))) s16x4*)pb0); \
    s16x4 hi = __builtin_amdgcn_ds_read_tr16_b64_v4i16((__attribute__((address_space(3))) s16x4*)pb1); \
    fb[n][ks] = __builtin_shufflevector(lo, hi, 0, 1, 2, 3, 4, 5, 6, 7); }
#define RA0(m) fa[m][0] = *(__attribute__((address_space(3))) s16x8*)(pa0 + 2048 * (m))
#define RA1(m) fa[m][1] = *(__attribute__((address_space(3))) s16x8*)(pa1 + 2048 * (m))

  for (int i = 0; i < 32; ++i) {
    __builtin_amdgcn_s_setprio(3);
    if (dopoll != 0ull)
      asm volatile("global_load_dwordx2 %0, %1, %2 sc0 sc1" : "=v"(pollv) : "v"(voffF), "s"(flg) : "memory");
    {
      asm volatile("" : "+v"(rdA), "+v"(rdB));
      lds_char* pa0 = (lds_char*)(size_t)rdA;
      lds_char* pa1 = (lds_char*)(size_t)(rdA ^ 64u);
      lds_char* pbb[4] = {(lds_char*)(size_t)rdB, (lds_char*)(size_t)(rdB ^ 32u),
                          (lds_char*)(size_t)(rdB ^ 64u), (lds_char*)(size_t)(rdB ^ 96u)};
      const unsigned la = lds0 + swrite + wg * 8192;
      const unsigned lb = lds0 + swrite + A_B + wg * 4096;
      SCHED();
      gldsb(voffB0, bbase, lb);
      MF(6, 0, 1); MF(6, 1, 1); RB(0, 0); RB(1, 0); SCHED();
      glds128(voffB0, bbase, lb + 1024);
      MF(6, 2, 1); MF(6, 3, 1); RB(2, 0); RB(3, 0); SCHED();
      gldsb(voffB1, bbase, lb + 2048);
      MF(7, 0, 1); MF(7, 1, 1); RA0(0); RA0(1); SCHED();
      glds128(voffB1, bbase, lb + 3072);
      MF(7, 2, 1); MF(7, 3, 1); RA0(2); RA0(3); SCHED();
      glds(voffA0, abase, la);
      RA0(4); RA0(5); RA0(6); SCHED();
      glds(voffA1, abase, la + 1024);
      RA0(7); RB(0, 1); RB(1, 1); SCHED();
      glds(voffA0, abase + 16384, la + 2048);
      RB(2, 1); RB(3, 1); RA1(0); SCHED();
      glds(voffA1, abase + 16384, la + 3072);
      RA1(1); RA1(2); RA1(3); SCHED();
      glds(voffA0, abase + 32768, la + 4096);
      RA1(4); RA1(5); SCHED();
      glds(voffA1, abase + 32768, la + 5120);
      RA1(6); RA1(7); SCHED();
      glds(voffA0, abase + 49152, la + 6144);
      glds(voffA1, abase + 49152, la + 7168);
      SCHED();
    }
    int pend = 0;
    if (i == myround - 1) { push(); pend = 2; }
    if (i == flagit && leader) { sendflag(); pend = 1; }
    WAITL0();
    __builtin_amdgcn_s_setprio(0);
    BAR();
    SCHED();
    unsigned ktv;
#pragma unroll
    for (int n = 0; n < 4; ++n) MF(0, n, 0);
    SCHED();
    asm volatile("v_mov_b32 %0, %1\n\tds_read_b32 %0, %0" : "=v"(ktv) : "s"(ldsw + (unsigned)((i & 1) * 8)) : "memory");
    SCHED();
#pragma unroll
    for (int m = 1; m < 5; ++m)
#pragma unroll
      for (int n = 0; n < 4; ++n) MF(m, n, 0);
    SCHED();
    {
      asm volatile("s_waitcnt lgkmcnt(0)" : "+v"(ktv) :: "memory");
      const unsigned kt = __builtin_amdgcn_readfirstlane(ktv) & 63u;
      abase = xl2 + (((kt >> 3) != (unsigned)rank) ? dstg : 0ull) + (size_t)(((kt >> 3) << 19) | ((kt & 7u) << 7));
      bbase = wcol + (size_t)(kt << 21);
      const unsigned nx = (swrite >= STAGE_B) ? swrite - STAGE_B : swrite + 2 * STAGE_B;
      rdA += swrite - sread; rdB += swrite - sread;
      sread = swrite; swrite = nx;
    }
#pragma unroll
    for (int m = 5; m < 8; ++m)
#pragma unroll
      for (int n = 0; n < 4; ++n) MF(m, n, 0);
#pragma unroll
    for (int k = 0; k < 12; ++k) { __builtin_amdgcn_sched_group_barrier(0x008, 1, 0); __builtin_amdgcn_sched_group_barrier(0x006, 2, 0); }
    SCHED();
    asm volatile("" : "+s"(abase), "+s"(bbase), "+s"(swrite), "+v"(rdA), "+v"(rdB));
    if (leader) {
      if (i >= 1 && i < 29) {
        unsigned long long rdy = rem & known;
        if (dopoll != 0ull) {
          if (pend == 0) WAITV(12); else if (pend == 1) WAITV(13); else WAITV(14);
          asm volatile("" : "+v"(pollv) :: "memory");
          unsigned long long bl = __builtin_amdgcn_ballot_w64(pollv[0] >= call && pollv[1] >= call);
          bl = bl & (bl >> 8) & 0x00FF00FF00FF00FFull; known |= bl | (bl << 8); rdy = known & rem;
          if (rdy == 0ull) {
            unsigned long long t0 = __builtin_amdgcn_s_memrealtime();
            for (;;) {
              asm volatile("global_load_dwordx2 %0, %1, %2 sc0 sc1\n\ts_waitcnt vmcnt(0)" : "=v"(pollv) : "v"(voffF), "s"(flg) : "memory");
              bl = __builtin_amdgcn_ballot_w64(pollv[0] >= call && pollv[1] >= call);
              bl = bl & (bl >> 8) & 0x00FF00FF00FF00FFull; known |= bl | (bl << 8); rdy = known & rem;
              if (rdy != 0ull) break;
              if (__builtin_amdgcn_readfirstlane(__hip_atomic_load(tmo, __ATOMIC_RELAXED, __HIP_MEMORY_SCOPE_AGENT)) == call ||
                  __builtin_amdgcn_s_memrealtime() - t0 > DEADLINE_TICKS) {
                __hip_atomic_store(tmo, call, __ATOMIC_RELAXED, __HIP_MEMORY_SCOPE_AGENT);
                rdy = rem; known = ~0ull;
                unsigned t0v;
                asm volatile("v_mov_b32 %0, %1\n\tds_write_b32 %0, %0 offset:16" : "=&v"(t0v) : "s"(ldsw) : "memory");
                break;
              }
              __builtin_amdgcn_s_sleep(1);
            }
            WAITV(0);
          }
        }
        const unsigned t = (unsigned)__builtin_ctzll(rdy);
        rem &= ~(1ull << t);
        ktn = (((unsigned)rank + t + 1u) & 7u) * 8u + (t >> 3);
      }
      unsigned t0v, t1v;
      asm volatile("v_mov_b32 %0, %2\n\tv_mov_b32 %1, %3\n\tds_write_b32 %0, %1" : "=&v"(t0v), "=&v"(t1v) : "s"(ldsw + (unsigned)((~i & 1) * 8)), "s"(ktn) : "memory");
      dopoll = rem & (0ull - rem) & ~known;
      asm volatile("" : "+s"(dopoll));
    }
    SCHED();
#pragma unroll
    for (int m = 0; m < 6; ++m)
#pragma unroll
      for (int n = 0; n < 4; ++n) MF(m, n, 1);
    SCHED();
    if (pend == 0) WAITV(0); else if (pend == 1) WAITV(1); else WAITV(2);
    BAR();
    SCHED();
  }
  __builtin_amdgcn_s_setprio(3);
#pragma unroll
  for (int m = 6; m < 8; ++m)
#pragma unroll
    for (int n = 0; n < 4; ++n) MF(m, n, 1);
  __builtin_amdgcn_s_setprio(0);
  SCHED();
  if (g == 0) BAR();

  WAITV(0);
  const bool bad = comm && (*(volatile __attribute__((address_space(3))) unsigned long long*)(lds + 3 * STAGE_B + 16) != 0ull);
  {
    lds_char* mine = lds + wv * 16384 + lane * 16;
#pragma unroll
    for (int m = 0; m < 4; ++m)
#pragma unroll
      for (int n = 0; n < 4; ++n) {
        f32x4 v = (g == 0) ? acc[m + 4][n] : acc[m][n];
        *(__attribute__((address_space(3))) f32x4*)(mine + (m * 4 + n) * 1024) = v;
      }
  }
  WAITL0();
  BAR();
  lds_char* theirs = lds + (wv ^ 4) * 16384;
  f32x4 fin[4][4];
#pragma unroll
  for (int m = 0; m < 4; ++m)
#pragma unroll
    for (int n = 0; n < 4; ++n) {
      f32x4 o = *(__attribute__((address_space(3))) f32x4*)(theirs + lane * 16 + (m * 4 + n) * 1024);
      f32x4 v = (g == 0) ? acc[m][n] : acc[m + 4][n];
      fin[m][n] = v + o;
    }
  WAITL0();
  SCHED();
#pragma unroll
  for (int m = 0; m < 4; ++m)
#pragma unroll
    for (int n = 0; n < 4; ++n) {
      f32x4 v = fin[m][n];
      bf16x2 lo = __builtin_convertvector(f32x2{silu_f(v[0]), silu_f(v[1])}, bf16x2);
      bf16x2 hi = __builtin_convertvector(f32x2{silu_f(v[2]), silu_f(v[3])}, bf16x2);
      u32x2 pk = u32x2{__builtin_bit_cast(unsigned, lo), __builtin_bit_cast(unsigned, hi)};
      if (bad) pk = u32x2{0x7fc07fc0u, 0x7fc07fc0u};
      const int row = 16 * m + fr, c = 2 * n + (fq >> 1);
      *(__attribute__((address_space(3))) u32x2*)(theirs + row * 128 + ((c ^ (row & 7)) * 16) + (fq & 1) * 8) = pk;
    }
  WAITL0();
  SCHED();
  {
    char* op = out + (size_t)(mh * 256 + wm * 128 + g * 64 + r8) * 32768 + (size_t)ntile * 256 + wn * 128 + (lane & 7) * 16;
#pragma unroll
    for (int i = 0; i < 8; ++i) {
      const int row = i * 8 + r8;
      u32x4 v = *(__attribute__((address_space(3))) u32x4*)(theirs + row * 128 + (((lane & 7) ^ (row & 7)) * 16));
      __builtin_nontemporal_store(v, (u32x4*)(op + (size_t)i * 8 * 32768));
    }
  }
  if (tid == 0) __hip_atomic_store((unsigned*)(ws + WS_CNT + bid * 4), call, __ATOMIC_RELAXED, __HIP_MEMORY_SCOPE_AGENT);
}

extern "C" void kernel_launch(int rank, int nranks, ncclComm_t comm,
                              hipStream_t stream,
                              void* const* d_in, const int* in_sizes, int n_in,
                              void* d_out, int out_size,
                              const int* in_dtypes, int out_dtype,
                              void* const* d_out_all,
                              void* const* d_ws_all, size_t ws_bytes) {
  if (nranks != NR || ws_bytes < WS_STG + NSLOT * SLOT_B) { fprintf(stderr, "unexpected configuration\n"); return; }
  static bool init = false;
  if (!init) {
    HIP_CHECK(hipFuncSetAttribute((const void*)a2a_gemm_silu, hipFuncAttributeMaxDynamicSharedMemorySize, LDS_B));
    init = true;
  }
  hipLaunchKernelGGL(a2a_gemm_silu, dim3(256), dim3(512), LDS_B, stream,
                     (const char*)d_in[0], (const char*)d_in[1], (char*)d_out,
                     (char* const*)d_ws_all, rank, 1);
  HIP_CHECK(hipGetLastError());
}
```
